```python
import numpy as np

kernel_name = "gpu_ag_gemm_m2048_k2048_n2048_bf16_silu"


ABSMAX_RTOL = 0.008

COMPUTE_DTYPE = "bfloat16"
EPILOGUE = "silu"
IN_QUANT = None
OUT_QUANT = None
IN_DTYPES = {"x": "bfloat16", "w_mat": "bfloat16"}
WORLD_SIZE = 8

def _to_bf16(x):
    u = np.ascontiguousarray(x, dtype=np.float32).view(np.uint32)
    return ((u + 0x7FFF + ((u >> 16) & 1)) & 0xFFFF0000).view(np.float32)

def setup_inputs():

    x = _to_bf16(np.random.randn(2048, 2048).astype(np.float32))
    w_mat = _to_bf16((np.random.randn(2048, 2048) / np.sqrt(2048)).astype(np.float32))
    return {"x": x, "w_mat": w_mat}

def _epilogue(y):
    return (y / (1.0 + np.exp(-np.clip(y, -60.0, 60.0)))).astype(np.float32)

def reference(x, w_mat):
    y = (x @ w_mat).astype(np.float32)
    return _epilogue(y)
```

```cpp
#include <hip/hip_runtime.h>
#include <rccl/rccl.h>
#include <cstdio>
#include <cstdint>

namespace {

constexpr int R = 8;
constexpr int ML = 256;
constexpr int KK = 2048;
constexpr int NL = 256;
#define NCH 8
constexpr int KC = KK / NCH;
constexpr unsigned SHARD_BYTES = ML * KK * 2;
constexpr unsigned SLOT_BYTES = R * SHARD_BYTES;
constexpr unsigned CTRL_OFF = 2 * SLOT_BYTES;
constexpr unsigned CHUNK_BYTES = ML * KC * 2;
constexpr unsigned PIECES = CHUNK_BYTES / 1024;
constexpr unsigned FLAG_OFF = CTRL_OFF;
constexpr unsigned FLAG_BYTES = R * NCH * PIECES * 4;
constexpr unsigned CNT_OFF = FLAG_OFF + FLAG_BYTES;
constexpr unsigned TMO_OFF = CNT_OFF + 256 * 128;
constexpr unsigned WS_NEED = TMO_OFF + 128;
constexpr unsigned long long DEADLINE = 50000000ull;

#define ST_AUX 17
#define LD_AUX 16
#define WIN 3

typedef unsigned u32x4 __attribute__((ext_vector_type(4)));
typedef unsigned u32x2 __attribute__((ext_vector_type(2)));
typedef float f32x4 __attribute__((ext_vector_type(4)));
typedef short bf16x8 __attribute__((ext_vector_type(8)));

struct Params {
  const void* x; const void* w; void* out;
  void* ws[R];
  int rank;
};

__device__ __forceinline__ __amdgpu_buffer_rsrc_t mk_rsrc(const void* p, unsigned bytes) {
  return __builtin_amdgcn_make_buffer_rsrc(const_cast<void*>(p), 0, bytes, 0x00020000);
}
template <int N> __device__ __forceinline__ void wait_vm() { asm volatile("s_waitcnt vmcnt(%0)" ::"n"(N) : "memory"); }

__device__ __forceinline__ unsigned pack_bf16(float lo, float hi) {
  unsigned r; asm("v_cvt_pk_bf16_f32 %0, %1, %2" : "=v"(r) : "v"(lo), "v"(hi)); return r;
}
__device__ __forceinline__ float silu(float y) {
  float t = fminf(fmaxf(y, -60.f), 60.f);
  float e = __builtin_amdgcn_exp2f(t * -1.44269504088896f);
  return y * __builtin_amdgcn_rcpf(1.f + e);
}

__global__ __launch_bounds__(512, 2) void ag_gemm_kernel(Params p) {
  __shared__ __attribute__((aligned(16))) float red[4 * 32 * 68];
  const int b = blockIdx.x;
  const int lane = threadIdx.x & 63;
  const int wave = __builtin_amdgcn_readfirstlane(threadIdx.x >> 6);
  const int rank = p.rank;
  char* myws = (char*)p.ws[rank];
  unsigned* cntp = (unsigned*)(myws + CNT_OFF + b * 128);
  const unsigned call = __builtin_amdgcn_readfirstlane(__hip_atomic_load(cntp, __ATOMIC_RELAXED, __HIP_MEMORY_SCOPE_AGENT)) + 1u;
  const unsigned slot = call & 1u;

  if (wave >= 4) {
    constexpr int SPC = (PIECES * 7) / (224 * 4);
    if (b >= 224) return;
    const int pw = wave - 4;
    const int pi = b % 7, j = b / 7;
    const int peer = (rank + 1 + pi) & 7;
    const int row0 = 8 * j + 2 * pw;
    auto xr = mk_rsrc(p.x, SHARD_BYTES);
    constexpr int NST = 8;
    u32x4 v[NST];
#pragma unroll
    for (int s = 0; s < NST; ++s) {
      unsigned off;
      if (NCH == 8) off = (row0 + (lane >> 5)) * 4096 + s * 512 + (lane & 31) * 16;
      else          off = (row0 + (s % SPC)) * 4096 + (s / SPC) * 1024 + lane * 16;
      v[s] = __builtin_amdgcn_raw_buffer_load_b128(xr, off, 0, 0);
    }
    char* pws = (char*)p.ws[peer];
    auto dr = mk_rsrc(pws + slot * SLOT_BYTES + rank * SHARD_BYTES, SHARD_BYTES);
    unsigned* flags = (unsigned*)(pws + FLAG_OFF) + rank * NCH * PIECES;
    wait_vm<0>();
    auto do_store = [&](int s) {
      const int c = s / SPC;
      const unsigned piece = (row0 * KC * 2) / 1024 + (s % SPC);
      __builtin_amdgcn_raw_buffer_store_b128(v[s], dr, piece * 1024 + lane * 16, c * CHUNK_BYTES, ST_AUX);
    };
    auto do_flag = [&](int s) {
      const int c = s / SPC;
      const unsigned piece = (row0 * KC * 2) / 1024 + (s % SPC);
      if (lane == 0) __hip_atomic_store(flags + c * PIECES + piece, call, __ATOMIC_RELAXED, __HIP_MEMORY_SCOPE_SYSTEM);
    };
#pragma unroll
    for (int s = 0; s < WIN && s < NST; ++s) do_store(s);
#pragma unroll
    for (int s = 0; s < NST; ++s) {
      constexpr int W = WIN;
      const int nf = (s < W) ? s : (W - 1);
      const int last_s = (s + W - 1 < NST - 1) ? (s + W - 1) : (NST - 1);
      const int ns = last_s - s;
      switch (nf + ns) {
        case 0: wait_vm<0>(); break; case 1: wait_vm<1>(); break; case 2: wait_vm<2>(); break; case 3: wait_vm<3>(); break;
        case 4: wait_vm<4>(); break; case 5: wait_vm<5>(); break; case 6: wait_vm<6>(); break; case 7: wait_vm<7>(); break;
        case 8: wait_vm<8>(); break; case 9: wait_vm<9>(); break; case 10: wait_vm<10>(); break; case 11: wait_vm<11>(); break;
        case 12: wait_vm<12>(); break; case 13: wait_vm<13>(); break; default: wait_vm<14>(); break;
      }
      do_flag(s);
      if (s + W < NST) do_store(s + W);
    }
    return;
  }

  const int w = wave;
  const int xcd = b & 7, idx = b >> 3;
  const int src = (rank + xcd) & 7;
  const int rb = idx >> 2, nt = idx & 3;
  const int l15 = lane & 15, q = lane >> 4;
  const bool local = (src == rank);
  const unsigned rs = local ? 4096u : (unsigned)(KC * 2);
  const unsigned cs = local ? (unsigned)(KC * 2) : CHUNK_BYTES;
  auto ar = mk_rsrc(local ? (const char*)p.x : (const char*)(myws + slot * SLOT_BYTES + src * SHARD_BYTES), SHARD_BYTES);
  auto wr = mk_rsrc(p.w, KK * NL * 2);
  const unsigned* flags = (const unsigned*)(myws + FLAG_OFF) + src * NCH * PIECES;
  unsigned* tmo = (unsigned*)(myws + TMO_OFF);
  constexpr int KW = KC / 4;
  constexpr int KS = KW / 32;
  constexpr int FPB = (32 * KC * 2) / 1024;
  f32x4 acc[2][4];
#pragma unroll
  for (int i = 0; i < 2; ++i)
#pragma unroll
    for (int j = 0; j < 4; ++j) acc[i][j] = (f32x4){0.f, 0.f, 0.f, 0.f};
  const unsigned a_off = (rb * 32 + l15) * rs + (w * KW + q * 8) * 2;
  const unsigned w_off = (w * KW + q * 8) * (NL * 2) + nt * 128 + l15 * 8;
  bool dead = false;

  for (int c = 0; c < NCH; ++c) {
    bf16x8 bf[KS][4];
#pragma unroll
    for (int ks = 0; ks < KS; ++ks) {
      u32x2 raw[8];
#pragma unroll
      for (int j = 0; j < 8; ++j)
        raw[j] = __builtin_amdgcn_raw_buffer_load_b64(wr, w_off + (ks * 32 + j) * (NL * 2), c * KC * NL * 2, 0);
      u32x4 f0, f1, f2, f3;
#pragma unroll
      for (int pp = 0; pp < 4; ++pp) {
        f0[pp] = __builtin_amdgcn_perm(raw[2 * pp + 1].x, raw[2 * pp].x, 0x05040100u);
        f1[pp] = __builtin_amdgcn_perm(raw[2 * pp + 1].x, raw[2 * pp].x, 0x07060302u);
        f2[pp] = __builtin_amdgcn_perm(raw[2 * pp + 1].y, raw[2 * pp].y, 0x05040100u);
        f3[pp] = __builtin_amdgcn_perm(raw[2 * pp + 1].y, raw[2 * pp].y, 0x07060302u);
      }
      bf[ks][0] = __builtin_bit_cast(bf16x8, f0); bf[ks][1] = __builtin_bit_cast(bf16x8, f1);
      bf[ks][2] = __builtin_bit_cast(bf16x8, f2); bf[ks][3] = __builtin_bit_cast(bf16x8, f3);
    }
    if (!local && !dead) {
      const unsigned* f = flags + c * PIECES + rb * FPB + (lane % FPB);
      unsigned long long t0 = __builtin_amdgcn_s_memrealtime();
      while (true) {
        unsigned v = __hip_atomic_load(f, __ATOMIC_RELAXED, __HIP_MEMORY_SCOPE_SYSTEM);
        if (__all((int)(v - call) >= 0)) break;
        if (__builtin_amdgcn_readfirstlane(__hip_atomic_load(tmo, __ATOMIC_RELAXED, __HIP_MEMORY_SCOPE_AGENT)) == call ||
            __builtin_amdgcn_s_memrealtime() - t0 > DEADLINE) {
          if (lane == 0) __hip_atomic_store(tmo, call, __ATOMIC_RELAXED, __HIP_MEMORY_SCOPE_AGENT);
          dead = true;
          break;
        }
        __builtin_amdgcn_s_sleep(2);
      }
      asm volatile("" ::: "memory");
    }
    bf16x8 af[KS][2];
#pragma unroll
    for (int ks = 0; ks < KS; ++ks)
#pragma unroll
      for (int mt = 0; mt < 2; ++mt) {
        u32x4 t = __builtin_amdgcn_raw_buffer_load_b128(ar, a_off + mt * 16 * rs + ks * 64, c * cs, LD_AUX);
        af[ks][mt] = __builtin_bit_cast(bf16x8, t);
      }
#pragma unroll
    for (int ks = 0; ks < KS; ++ks)
#pragma unroll
      for (int mt = 0; mt < 2; ++mt)
#pragma unroll
        for (int i = 0; i < 4; ++i)
          acc[mt][i] = __builtin_amdgcn_mfma_f32_16x16x32_bf16(bf[ks][i], af[ks][mt], acc[mt][i], 0, 0, 0);
  }

#pragma unroll
  for (int mt = 0; mt < 2; ++mt)
#pragma unroll
    for (int r = 0; r < 4; ++r) {
      f32x4 v = {acc[mt][0][r], acc[mt][1][r], acc[mt][2][r], acc[mt][3][r]};
      *(f32x4*)&red[(w * 32 + mt * 16 + l15) * 68 + q * 16 + r * 4] = v;
    }
  __syncthreads();
  {
    const int t = threadIdx.x;
    const int row = t >> 3, cg = t & 7;
    f32x4 s0 = {0.f, 0.f, 0.f, 0.f}, s1 = {0.f, 0.f, 0.f, 0.f};
#pragma unroll
    for (int ww = 0; ww < 4; ++ww) {
      s0 += *(const f32x4*)&red[(ww * 32 + row) * 68 + cg * 8];
      s1 += *(const f32x4*)&red[(ww * 32 + row) * 68 + cg * 8 + 4];
    }
    const bool bad = __builtin_amdgcn_readfirstlane(__hip_atomic_load(tmo, __ATOMIC_RELAXED, __HIP_MEMORY_SCOPE_AGENT)) == call;
    u32x4 o;
    o.x = pack_bf16(silu(s0[0]), silu(s0[1]));
    o.y = pack_bf16(silu(s0[2]), silu(s0[3]));
    o.z = pack_bf16(silu(s1[0]), silu(s1[1]));
    o.w = pack_bf16(silu(s1[2]), silu(s1[3]));
    if (bad) o = (u32x4){0x7fc07fc0u, 0x7fc07fc0u, 0x7fc07fc0u, 0x7fc07fc0u};
    char* op = (char*)p.out + (size_t)(src * ML + rb * 32 + row) * (NL * 2) + nt * 128 + cg * 16;
    *(u32x4*)op = o;
    if (t == 0) __hip_atomic_store(cntp, call, __ATOMIC_RELAXED, __HIP_MEMORY_SCOPE_AGENT);
  }
}

}

extern "C" void kernel_launch(int rank, int nranks, ncclComm_t comm, hipStream_t stream,
                              void* const* d_in, const int* in_sizes, int n_in,
                              void* d_out, int out_size, const int* in_dtypes, int out_dtype,
                              void* const* d_out_all, void* const* d_ws_all, size_t ws_bytes) {
  if (nranks != R || (unsigned)rank >= (unsigned)R || ws_bytes < WS_NEED || n_in < 2 ||
      in_sizes[0] != ML * KK || in_sizes[1] != KK * NL || out_size != R * ML * NL) {
    fprintf(stderr, "kernel_launch: unexpected shapes\n");
    return;
  }
  Params p;
  p.x = d_in[0]; p.w = d_in[1]; p.out = d_out; p.rank = rank;
  for (int i = 0; i < R; ++i) p.ws[i] = d_ws_all[i];
  ag_gemm_kernel<<<256, 512, 0, stream>>>(p);
}
```

```cpp
#include <hip/hip_runtime.h>
#include <rccl/rccl.h>
#include <cstdio>
#include <cstdint>

namespace {

constexpr int R = 8;
constexpr int ML = 256;
constexpr int KK = 2048;
constexpr int NL = 256;
constexpr int NCH = 8;
constexpr int KC = KK / NCH;
constexpr unsigned SHARD_BYTES = ML * KK * 2;
constexpr unsigned SRC_BYTES = 2 * SHARD_BYTES;
constexpr unsigned CTRL_OFF = R * SRC_BYTES;
constexpr unsigned CHUNK_BYTES = ML * KC * 2;
constexpr unsigned PIECES = CHUNK_BYTES / 1024;
constexpr unsigned FLAG_OFF = CTRL_OFF;
constexpr unsigned FLAG_BYTES = R * NCH * PIECES * 4;
constexpr unsigned CNT_OFF = FLAG_OFF + FLAG_BYTES;
constexpr unsigned TMO_OFF = CNT_OFF + 256 * 128;
constexpr unsigned DBG_OFF = CTRL_OFF + 512 * 1024;
constexpr unsigned long long DEADLINE = 50000000ull;

#define WIN 3
#define LD_AUX 16

typedef unsigned u32x4 __attribute__((ext_vector_type(4)));
typedef unsigned u32x2 __attribute__((ext_vector_type(2)));
typedef float f32x4 __attribute__((ext_vector_type(4)));
typedef short bf16x8 __attribute__((ext_vector_type(8)));

struct Params {
  const void* x; const void* w; void* out;
  void* ws[R];
  int rank;
};

__device__ __forceinline__ __amdgpu_buffer_rsrc_t mk_rsrc(const void* p, unsigned bytes) {
  return __builtin_amdgcn_make_buffer_rsrc(const_cast<void*>(p), 0, bytes, 0x00020000);
}
__device__ __forceinline__ void wait_vm_n(int n) {
  switch (n) {
#define C(N) case N: asm volatile("s_waitcnt vmcnt(" #N ")" ::: "memory"); break;
    C(0) C(1) C(2) C(3) C(4) C(5) C(6) C(7) C(8) C(9) C(10) C(11) C(12) C(13) C(14) C(15) C(16) C(17) C(18) C(19) C(20)
#undef C
    default: asm volatile("s_waitcnt vmcnt(0)" ::: "memory"); break;
  }
}
__device__ __forceinline__ unsigned pack_bf16(float lo, float hi) {
  unsigned r; asm("v_cvt_pk_bf16_f32 %0, %1, %2" : "=v"(r) : "v"(lo), "v"(hi)); return r;
}
__device__ __forceinline__ float silu(float y) {
  float t = fminf(fmaxf(y, -60.f), 60.f);
  float e = __builtin_amdgcn_exp2f(t * -1.44269504088896f);
  return y * __builtin_amdgcn_rcpf(1.f + e);
}
struct BRaw { u32x2 r[8]; };
__device__ __forceinline__ void load_braw(BRaw& b, __amdgpu_buffer_rsrc_t wr, unsigned voff, unsigned soff) {
#pragma unroll
  for (int j = 0; j < 8; ++j) b.r[j] = __builtin_amdgcn_raw_buffer_load_b64(wr, voff + j * (NL * 2), soff, 0);
}
__device__ __forceinline__ void perm_b(bf16x8 (&f)[4], const BRaw& b) {
  u32x4 f0, f1, f2, f3;
#pragma unroll
  for (int pp = 0; pp < 4; ++pp) {
    f0[pp] = __builtin_amdgcn_perm(b.r[2 * pp + 1].x, b.r[2 * pp].x, 0x05040100u);
    f1[pp] = __builtin_amdgcn_perm(b.r[2 * pp + 1].x, b.r[2 * pp].x, 0x07060302u);
    f2[pp] = __builtin_amdgcn_perm(b.r[2 * pp + 1].y, b.r[2 * pp].y, 0x05040100u);
    f3[pp] = __builtin_amdgcn_perm(b.r[2 * pp + 1].y, b.r[2 * pp].y, 0x07060302u);
  }
  f[0] = __builtin_bit_cast(bf16x8, f0); f[1] = __builtin_bit_cast(bf16x8, f1);
  f[2] = __builtin_bit_cast(bf16x8, f2); f[3] = __builtin_bit_cast(bf16x8, f3);
}

constexpr int RED_LD = 68;
constexpr int RED_MAIN = 4 * 32 * RED_LD;
constexpr int RED_LOC = 8 * 16 * RED_LD;

__global__ __launch_bounds__(512, 2) void ag_gemm_kernel(Params p) {
  __shared__ __attribute__((aligned(16))) float red[RED_MAIN + RED_LOC + 16];
  volatile unsigned* lflag = (volatile unsigned*)&red[RED_MAIN + RED_LOC];
  const int b = blockIdx.x;
  const int lane = threadIdx.x & 63;
  const int wave = __builtin_amdgcn_readfirstlane(threadIdx.x >> 6);
  const int rank = p.rank;
  char* myws = (char*)p.ws[rank];
  unsigned* cntp = (unsigned*)(myws + CNT_OFF + b * 128);

  if (b < 32) {
    if (wave < 4) return;
    const int pw = wave - 4;
    const int g0 = b * 4 + pw;
    auto xr = mk_rsrc(p.x, SHARD_BYTES);
    u32x4 v[NCH];
    const unsigned xoff = (2 * g0 + (lane >> 5)) * 4096 + (lane & 31) * 16;
    v[0] = __builtin_amdgcn_raw_buffer_load_b128(xr, xoff, 0, 0);
    cntp += pw;
    const unsigned cv = __hip_atomic_load(cntp, __ATOMIC_RELAXED, __HIP_MEMORY_SCOPE_AGENT);
#pragma unroll
    for (int c = 1; c < NCH; ++c) v[c] = __builtin_amdgcn_raw_buffer_load_b128(xr, xoff + c * 512, 0, 0);
    const unsigned call = __builtin_amdgcn_readfirstlane(cv) + 1u;
    const unsigned slot = call & 1u;
    const int rot = g0 % 7;
    __amdgpu_buffer_rsrc_t dr[7];
#pragma unroll
    for (int i = 0; i < 7; ++i) {
      int pi = i + rot; if (pi >= 7) pi -= 7;
      const int peer = (rank + 1 + pi) & 7;
      dr[i] = mk_rsrc((char*)p.ws[peer] + rank * SRC_BYTES + slot * SHARD_BYTES, SHARD_BYTES);
    }
    char* fpeer;
    {
      const int peer = (rank + 1 + (lane % 7)) & 7;
      fpeer = (char*)p.ws[0];
#pragma unroll
      for (int i = 1; i < R; ++i) fpeer = (peer == i) ? (char*)p.ws[i] : fpeer;
    }
    unsigned* fptr = (unsigned*)(fpeer + FLAG_OFF) + rank * NCH * PIECES + g0;
    constexpr int SPC = 7, NS = NCH * SPC;
    const unsigned doff = g0 * 1024 + lane * 16;
#pragma unroll
    for (int s = 0; s < NS + WIN; ++s) {
      if (s >= WIN) {
        int n = 0;
        for (int t = s - WIN + 1; t < s; ++t) {
          if (t < NS) ++n;
          if (t - WIN >= 0 && (t - WIN) % SPC == SPC - 1) ++n;
        }
        wait_vm_n(n);
        if ((s - WIN) % SPC == SPC - 1) {
          const int c = (s - WIN) / SPC;
          if (lane < SPC) __hip_atomic_store(fptr + c * PIECES, call, __ATOMIC_RELAXED, __HIP_MEMORY_SCOPE_SYSTEM);
        }
      }
      if (s < NS) __builtin_amdgcn_raw_buffer_store_b128(v[s / SPC], dr[s % SPC], doff, (s / SPC) * CHUNK_BYTES, 17);
    }
    if (lane == 0) __hip_atomic_store(cntp, call, __ATOMIC_RELAXED, __HIP_MEMORY_SCOPE_AGENT);
    return;
  }

  const int rb = b & 7, idx = (b >> 3) - 4, nt = idx & 3, srcidx = 1 + (idx >> 2);
  const int src = (rank + srcidx) & 7;
  const bool ext = srcidx <= 2;
  const int l15 = lane & 15, q = lane >> 4;
  const int mh = wave & 1, kq = wave >> 1;
  if (threadIdx.x < NCH) lflag[threadIdx.x] = 0;
  const unsigned cv = __hip_atomic_load(cntp, __ATOMIC_RELAXED, __HIP_MEMORY_SCOPE_AGENT);
  auto wr = mk_rsrc(p.w, KK * NL * 2);
  auto xr = mk_rsrc(p.x, ext ? SHARD_BYTES : 0u);
  const unsigned w_off = (kq * 64 + q * 8) * (NL * 2) + nt * 128 + l15 * 8;
  const unsigned xl_off = (rb * 32 + (srcidx - 1) * 16 + l15) * 4096 + (kq * 64 + mh * 32 + q * 8) * 2;
  BRaw braw[NCH][2];
  u32x4 xl[NCH];
#pragma unroll
  for (int c = 0; c < 2; ++c) {
    load_braw(braw[c][0], wr, w_off, c * KC * NL * 2);
    load_braw(braw[c][1], wr, w_off + 32 * (NL * 2), c * KC * NL * 2);
    xl[c] = __builtin_amdgcn_raw_buffer_load_b128(xr, xl_off, c * (KC * 2), 0);
  }
  const unsigned call = __builtin_amdgcn_readfirstlane(cv) + 1u;
  const unsigned slot = call & 1u;
  unsigned* tmo = (unsigned*)(myws + TMO_OFF);
  auto ar = mk_rsrc(myws + src * SRC_BYTES + slot * SHARD_BYTES, SHARD_BYTES);
  const unsigned* flags = (const unsigned*)(myws + FLAG_OFF) + src * NCH * PIECES + rb * 16 + l15;
  const unsigned a_off = (rb * 32 + mh * 16 + l15) * (KC * 2) + (kq * 64 + q * 8) * 2;
  f32x4 acc[4], accl[4];
#pragma unroll
  for (int j = 0; j < 4; ++j) { acc[j] = (f32x4){0.f, 0.f, 0.f, 0.f}; accl[j] = (f32x4){0.f, 0.f, 0.f, 0.f}; }
  asm volatile("s_waitcnt lgkmcnt(0)" ::: "memory");
  __builtin_amdgcn_s_barrier();
  bool dead = false;
  unsigned tmov = 0;
#pragma unroll
  for (int c = 0; c < NCH; ++c) {
    if (c + 2 < NCH) {
      load_braw(braw[c + 2][0], wr, w_off, (c + 2) * KC * NL * 2);
      load_braw(braw[c + 2][1], wr, w_off + 32 * (NL * 2), (c + 2) * KC * NL * 2);
      xl[c + 2] = __builtin_amdgcn_raw_buffer_load_b128(xr, xl_off, (c + 2) * (KC * 2), 0);
    }
    bf16x8 bf[2][4];
    perm_b(bf[0], braw[c][0]);
    perm_b(bf[1], braw[c][1]);
    if (ext) {
      const bf16x8 xf = __builtin_bit_cast(bf16x8, xl[c]);
#pragma unroll
      for (int i = 0; i < 4; ++i) accl[i] = __builtin_amdgcn_mfma_f32_16x16x32_bf16(mh ? bf[1][i] : bf[0][i], xf, accl[i], 0, 0, 0);
      if (c == NCH - 1) {
        float* rl = red + RED_MAIN;
#pragma unroll
        for (int r = 0; r < 4; ++r) {
          f32x4 v = {accl[0][r], accl[1][r], accl[2][r], accl[3][r]};
          *(f32x4*)&rl[(wave * 16 + l15) * RED_LD + q * 16 + r * 4] = v;
        }
        __syncthreads();
        if (threadIdx.x < 256) {
          const int t = threadIdx.x, row = t >> 4, cg = t & 15;
          f32x4 s = {0.f, 0.f, 0.f, 0.f};
#pragma unroll
          for (int ww = 0; ww < 8; ++ww) s += *(const f32x4*)&rl[(ww * 16 + row) * RED_LD + cg * 4];
          u32x2 o;
          o.x = pack_bf16(silu(s[0]), silu(s[1]));
          o.y = pack_bf16(silu(s[2]), silu(s[3]));
          char* op = (char*)p.out + (size_t)(rank * ML + rb * 32 + (srcidx - 1) * 16 + row) * (NL * 2) + nt * 128 + cg * 8;
          *(u32x2*)op = o;
        }
      }
    }
#pragma unroll
    for (int ks = 0; ks < 2; ++ks)
#pragma unroll
      for (int i = 0; i < 4; ++i) asm volatile("" : "+v"(bf[ks][i]));
    if (wave == 0) {
      if (!dead) {
        unsigned long long t0 = __builtin_amdgcn_s_memrealtime();
        while (true) {
          unsigned v = __hip_atomic_load(flags + c * PIECES, __ATOMIC_RELAXED, __HIP_MEMORY_SCOPE_SYSTEM);
          if (__all((int)(v - call) >= 0)) break;
          if (__builtin_amdgcn_readfirstlane(__hip_atomic_load(tmo, __ATOMIC_RELAXED, __HIP_MEMORY_SCOPE_AGENT)) == call ||
              __builtin_amdgcn_s_memrealtime() - t0 > DEADLINE) {
            if (lane == 0) __hip_atomic_store(tmo, call, __ATOMIC_RELAXED, __HIP_MEMORY_SCOPE_AGENT);
            dead = true;
            break;
          }
          __builtin_amdgcn_s_sleep(1);
        }
      }
      asm volatile("" ::: "memory");
      if (lane == 0) lflag[c] = 1;
    } else {
      while (lflag[c] == 0) __builtin_amdgcn_s_sleep(1);
      asm volatile("" ::: "memory");
    }
    bf16x8 af[2];
#pragma unroll
    for (int ks = 0; ks < 2; ++ks)
      af[ks] = __builtin_bit_cast(bf16x8, __builtin_amdgcn_raw_buffer_load_b128(ar, a_off + ks * 64, c * CHUNK_BYTES, LD_AUX));
    if (c == NCH - 1) tmov = __hip_atomic_load(tmo, __ATOMIC_RELAXED, __HIP_MEMORY_SCOPE_AGENT);
#pragma unroll
    for (int ks = 0; ks < 2; ++ks)
#pragma unroll
      for (int i = 0; i < 4; ++i)
        acc[i] = __builtin_amdgcn_mfma_f32_16x16x32_bf16(bf[ks][i], af[ks], acc[i], 0, 0, 0);
  }
#pragma unroll
  for (int r = 0; r < 4; ++r) {
    f32x4 v = {acc[0][r], acc[1][r], acc[2][r], acc[3][r]};
    *(f32x4*)&red[(kq * 32 + mh * 16 + l15) * RED_LD + q * 16 + r * 4] = v;
  }
  __syncthreads();
  {
    const bool bad = __builtin_amdgcn_readfirstlane(tmov) == call;
    const int t = threadIdx.x;
    const int row = t >> 4, cg = t & 15;
    f32x4 s = {0.f, 0.f, 0.f, 0.f};
#pragma unroll
    for (int ww = 0; ww < 4; ++ww) s += *(const f32x4*)&red[(ww * 32 + row) * RED_LD + cg * 4];
    u32x2 o;
    o.x = pack_bf16(silu(s[0]), silu(s[1]));
    o.y = pack_bf16(silu(s[2]), silu(s[3]));
    if (bad) o = (u32x2){0x7fc07fc0u, 0x7fc07fc0u};
    char* op = (char*)p.out + (size_t)(src * ML + rb * 32 + row) * (NL * 2) + nt * 128 + cg * 8;
    *(u32x2*)op = o;
  }
  if (threadIdx.x == 0) __hip_atomic_store(cntp, call, __ATOMIC_RELAXED, __HIP_MEMORY_SCOPE_AGENT);
}

}

extern "C" void kernel_launch(int rank, int nranks, ncclComm_t comm, hipStream_t stream,
                              void* const* d_in, const int* in_sizes, int n_in,
                              void* d_out, int out_size, const int* in_dtypes, int out_dtype,
                              void* const* d_out_all, void* const* d_ws_all, size_t ws_bytes) {
  if (nranks != R || (unsigned)rank >= (unsigned)R || ws_bytes < CTRL_OFF + (1u << 20) || n_in < 2 ||
      in_sizes[0] != ML * KK || in_sizes[1] != KK * NL || out_size != R * ML * NL) {
    fprintf(stderr, "kernel_launch: unexpected shapes\n");
    return;
  }
  Params p;
  p.x = d_in[0]; p.w = d_in[1]; p.out = d_out; p.rank = rank;
  for (int i = 0; i < R; ++i) p.ws[i] = d_ws_all[i];
  ag_gemm_kernel<<<256, 512, 0, stream>>>(p);
}
```

```cpp
#include <hip/hip_runtime.h>
#include <rccl/rccl.h>
#include <cstdio>
#include <cstdint>

namespace {

constexpr int R = 8, ML = 256, KK = 2048, NL = 256;
constexpr int NCH = 8, KC = KK / NCH;
constexpr int GMAJ = 10, GMIN = 5;
constexpr int ROW_MIN = GMAJ * 16, ROW_ALL = (GMAJ + GMIN) * 16;
constexpr unsigned XMAJ_BYTES = GMAJ * 65536;
constexpr unsigned XMIN_BYTES = GMIN * 65536;
constexpr unsigned XS_OFF = 0;
constexpr unsigned XO_OFF = 3 * XMAJ_BYTES;
constexpr unsigned XA_OFF = XO_OFF + 4 * XMIN_BYTES;
constexpr unsigned WS_OFF = XA_OFF + 8 * 65536;
constexpr unsigned OS_OFF = WS_OFF + 4 * 262144;
constexpr unsigned USED_BYTES = OS_OFF + 4 * 131072;
constexpr unsigned SLOT_BYTES = 6u << 20;
constexpr unsigned CTRL_OFF = 16u << 20;
constexpr unsigned CNT_OFF = CTRL_OFF;
constexpr unsigned TMO_OFF = CNT_OFF + 256 * 128;
constexpr unsigned TICK_OFF = TMO_OFF + 128;
constexpr unsigned CRED_OFF = TICK_OFF + 128;
constexpr unsigned DBG_OFF = CTRL_OFF + 512 * 1024;
constexpr unsigned long long DEADLINE = 50000000ull;
constexpr int NPUSH = 16;
constexpr int NZB = 82;
constexpr int ZPW = 8;

#define WINS 4
#define WINC 5
#define PACE 120
#define T_B 450u
#define T_MIN 550u
#define T_C 950u
#define T_ALL 1150u
#define STAG 100
#define APF 3
#define FIRST_SLEEP 2
#define POLL_SLEEP 2

typedef unsigned u32x4 __attribute__((ext_vector_type(4)));
typedef unsigned u32x2 __attribute__((ext_vector_type(2)));
typedef float f32x4 __attribute__((ext_vector_type(4)));
typedef short bf16x8 __attribute__((ext_vector_type(8)));

struct Params {
  const void* x; const void* w; void* out;
  void* ws[R];
  int rank; int pad;
  unsigned tab[256];
};
enum { K_TILE = 0, K_PUSH = 1, K_COLL = 2, K_IDLE = 3 };

__device__ __forceinline__ __amdgpu_buffer_rsrc_t mk_rsrc(const void* p, unsigned bytes) {
  return __builtin_amdgcn_make_buffer_rsrc(const_cast<void*>(p), 0, bytes, 0x00020000);
}
template <int AUX>
__device__ __forceinline__ void st128(u32x4 v, __amdgpu_buffer_rsrc_t r, unsigned voff, unsigned soff) {
  __builtin_amdgcn_raw_buffer_store_b128(v, r, voff, soff, AUX);
  asm volatile("s_nop 1" ::"v"(v));
}
__device__ __forceinline__ void wait_vm_n(int n) {
  switch (n) {
#define C(N) case N: asm volatile("s_waitcnt vmcnt(" #N ")" ::: "memory"); break;
    C(0) C(1) C(2) C(3) C(4) C(5) C(6) C(7) C(8) C(9) C(10) C(11) C(12) C(13) C(14) C(15) C(16) C(17) C(18) C(19) C(20) C(21) C(22) C(23) C(24)
#undef C
    default: asm volatile("s_waitcnt vmcnt(0)" ::: "memory"); break;
  }
}
__device__ __forceinline__ unsigned pack_bf16(float lo, float hi) {
  unsigned r; asm("v_cvt_pk_bf16_f32 %0, %1, %2" : "=v"(r) : "v"(lo), "v"(hi)); return r;
}
__device__ __forceinline__ float silu(float y) {
  float t = fminf(fmaxf(y, -60.f), 60.f);
  float e = __builtin_amdgcn_exp2f(t * -1.44269504088896f);
  return y * __builtin_amdgcn_rcpf(1.f + e);
}
__device__ __forceinline__ unsigned nz(unsigned v) { return v == 0u ? 0x80000000u : v; }
__device__ __forceinline__ u32x4 nz4(u32x4 v) { v.x = nz(v.x); v.y = nz(v.y); v.z = nz(v.z); v.w = nz(v.w); return v; }
__device__ __forceinline__ unsigned umin(unsigned a, unsigned b) { return a < b ? a : b; }
__device__ __forceinline__ unsigned umin4(u32x4 v) { return umin(umin(v.x, v.y), umin(v.z, v.w)); }

__device__ __forceinline__ void decode_pair(int pi, int mh, int& same, int& idx, int& g) {
  if (pi < 16) { same = 1; idx = pi & 3; g = 2 * (pi >> 2) + mh; }
  else if (pi < 24) { same = 0; idx = pi & 3; g = 2 * ((pi - 16) >> 2) + mh; }
  else if (pi < 26) { same = 0; idx = 2 * (pi - 24) + mh; g = 4; }
  else if (pi < 30) { same = 1; idx = pi - 26; g = 8 + mh; }
  else { same = 2; idx = 2 * (pi - 30) + mh; g = 0; }
}
__device__ __forceinline__ int global_row(int grp, int same, int idx, int g) {
  return same == 2 ? idx * ML + ROW_ALL : (same ? (grp * 4 + idx) * ML + 16 * g : ((grp ^ 1) * 4 + idx) * ML + ROW_MIN + 16 * g);
}

__device__ __forceinline__ void pace(unsigned long long t_start, unsigned ticks) {
  while ((unsigned)(__builtin_amdgcn_s_memrealtime() - t_start) < ticks) __builtin_amdgcn_s_sleep(1);
}
struct Waiter {
  unsigned* tmo; unsigned call; unsigned it; unsigned long long t0; bool dead;
  __device__ __forceinline__ void tick(int lane) {
    if ((++it & 63u) == 0u) {
      const unsigned long long now = __builtin_amdgcn_s_memrealtime();
      if (t0 == 0ull) t0 = now;
      if (__builtin_amdgcn_readfirstlane(__hip_atomic_load(tmo, __ATOMIC_RELAXED, __HIP_MEMORY_SCOPE_AGENT)) == call || now - t0 > DEADLINE) {
        if (lane == 0) __hip_atomic_store(tmo, call, __ATOMIC_RELAXED, __HIP_MEMORY_SCOPE_AGENT);
        dead = true;
      }
    }
  }
  __device__ __forceinline__ void reset() { it = 0; t0 = 0ull; }
};

struct BRaw { u32x2 r[8]; };
__device__ __forceinline__ void perm_b(bf16x8 (&f)[4], const BRaw& b) {
  u32x4 f0, f1, f2, f3;
#pragma unroll
  for (int pp = 0; pp < 4; ++pp) {
    f0[pp] = __builtin_amdgcn_perm(b.r[2 * pp + 1].x, b.r[2 * pp].x, 0x05040100u);
    f1[pp] = __builtin_amdgcn_perm(b.r[2 * pp + 1].x, b.r[2 * pp].x, 0x07060302u);
    f2[pp] = __builtin_amdgcn_perm(b.r[2 * pp + 1].y, b.r[2 * pp].y, 0x05040100u);
    f3[pp] = __builtin_amdgcn_perm(b.r[2 * pp + 1].y, b.r[2 * pp].y, 0x07060302u);
  }
  f[0] = __builtin_bit_cast(bf16x8, f0); f[1] = __builtin_bit_cast(bf16x8, f1);
  f[2] = __builtin_bit_cast(bf16x8, f2); f[3] = __builtin_bit_cast(bf16x8, f3);
}
__device__ __forceinline__ unsigned bmin(const BRaw& b) {
  unsigned m = 0xffffffffu;
#pragma unroll
  for (int j = 0; j < 8; ++j) m = umin(m, umin(b.r[j].x, b.r[j].y));
  return m;
}

constexpr int RED_LD = 68;
constexpr int RED_MAIN = 8 * 32 * RED_LD;

template <int NP, int NB>
__device__ __forceinline__ void collect(const Params& p, char* slotp, int hi, int pp0, int ppstep, int grp, int lane, Waiter& wt) {
  auto sr = mk_rsrc(slotp + OS_OFF + hi * 131072, 131072);
  auto orr = mk_rsrc(p.out, R * ML * NL * 2);
#pragma unroll 1
  for (int i0 = 0; i0 < NP; i0 += NB) {
    u32x4 v[NB]; unsigned tm;
    wt.reset();
    while (true) {
      asm volatile("" ::: "memory");
      unsigned m = 0xffffffffu;
#pragma unroll
      for (int b = 0; b < NB; ++b) {
        const int pp = pp0 + (i0 + b) * ppstep;
        v[b] = __builtin_amdgcn_raw_buffer_load_b128(sr, lane * 16, (pp < 120 ? pp : pp0) * 1024, 16);
      }
      tm = __hip_atomic_load(wt.tmo, __ATOMIC_RELAXED, __HIP_MEMORY_SCOPE_AGENT);
#pragma unroll
      for (int b = 0; b < NB; ++b) m = umin(m, umin4(v[b]));
      if (__all(m != 0u) || wt.dead) break;
      wt.tick(lane);
      __builtin_amdgcn_s_sleep(1);
    }
    const bool bad = wt.dead || __builtin_amdgcn_readfirstlane(tm) == wt.call;
#pragma unroll
    for (int b = 0; b < NB; ++b) {
      const int pp = pp0 + (i0 + b) * ppstep;
      if (pp < 120) {
        if (bad) v[b] = (u32x4){0x7fc07fc0u, 0x7fc07fc0u, 0x7fc07fc0u, 0x7fc07fc0u};
        int same, idx, g;
        decode_pair(pp >> 2, (pp >> 1) & 1, same, idx, g);
        const int row = global_row(grp ^ 1, same, idx, g) + (pp & 1) * 8;
        st128<0>(v[b], orr, (lane >> 3) * (NL * 2) + (lane & 7) * 16, row * (NL * 2) + hi * 128);
      }
    }
  }
}

__global__ __launch_bounds__(512, 2) void ag_gemm_kernel(Params p) {
  __shared__ __attribute__((aligned(16))) float red[RED_MAIN];
  const int b = blockIdx.x;
  const int lane = threadIdx.x & 63;
  const int wave = __builtin_amdgcn_readfirstlane(threadIdx.x >> 6);
  const int rank = p.rank, grp = rank >> 2, ri = rank & 3;
  char* myws = (char*)p.ws[rank];
  unsigned* cntp = (unsigned*)(myws + CNT_OFF + b * 128);
  const unsigned long long t_entry0 = __builtin_amdgcn_s_memrealtime();
  const unsigned tab = p.tab[b];
  const int kind = tab >> 20;
  unsigned* tmo = (unsigned*)(myws + TMO_OFF);

  if (kind == K_IDLE) return;
  if (kind != K_TILE) {
    cntp += wave;
    if (kind == K_COLL) {
      const unsigned call = __builtin_amdgcn_readfirstlane(__hip_atomic_load(cntp, __ATOMIC_RELAXED, __HIP_MEMORY_SCOPE_AGENT)) + 1u;
      Waiter wt{tmo, call, 0u, 0ull, false};
      const int dv = (b - NPUSH) * 8 + wave;
      collect<8, 1>(p, myws + (call & 1u) * SLOT_BYTES, dv & 3, dv >> 2, 8, grp, lane, wt);
      if (lane == 0) __hip_atomic_store(cntp, call, __ATOMIC_RELAXED, __HIP_MEMORY_SCOPE_AGENT);
      return;
    }
    const int v = b * 4 + (wave & 3);
    const bool swave = wave < 4;
    auto xr = mk_rsrc(p.x, ML * KK * 2);
    const unsigned lx = (lane & 15) * 4096 + (lane >> 4) * 16;
    const unsigned cv = __hip_atomic_load(cntp, __ATOMIC_RELAXED, __HIP_MEMORY_SCOPE_AGENT);
    const unsigned* credp = (const unsigned*)(myws + CRED_OFF) + ((rank + 1 + (lane % 7)) & 7) * 32;
    unsigned cred = __hip_atomic_load(credp, __ATOMIC_RELAXED, __HIP_MEMORY_SCOPE_SYSTEM);
    const unsigned stag = STAG * (v * PACE) / 6400;
    unsigned call, slot;
    auto get_call = [&]() -> bool {
      call = __builtin_amdgcn_readfirstlane(cv) + 1u;
      slot = call & 1u;
      if (!__all((int)(cred - (call - 1u)) >= 0)) {
        Waiter w2{tmo, call, 0u, 0ull, false};
        while (true) {
          __builtin_amdgcn_s_sleep(2);
          cred = __hip_atomic_load(credp, __ATOMIC_RELAXED, __HIP_MEMORY_SCOPE_SYSTEM);
          if (__all((int)(cred - (call - 1u)) >= 0)) break;
          w2.tick(lane);
          if (w2.dead) { if (lane == 0) __hip_atomic_store(cntp, call, __ATOMIC_RELAXED, __HIP_MEMORY_SCOPE_AGENT); return false; }
        }
      }
      return true;
    };
    if (swave) {
      constexpr int NRS = GMAJ + 1;
      u32x4 d[NRS]; unsigned doff[NRS];
      auto ld = [&](int r) {
        if (r < GMAJ) {
          const int g0 = r < 4 ? 0 : (r < 8 ? 4 : 8), ng = r < 8 ? 4 : 2, rr = r - g0;
          const int e = rr * 64 + v, c = e / (8 * ng), rem = e % (8 * ng);
          const int g = g0 + (rem >> 3), w = rem & 7;
          d[r] = __builtin_amdgcn_raw_buffer_load_b128(xr, lx, g * 16 * 4096 + c * 512 + w * 64, 0);
          doff[r] = ((c * GMAJ + g) * 8 + w) * 1024;
        } else {
          d[r] = __builtin_amdgcn_raw_buffer_load_b128(xr, lx, ROW_ALL * 4096 + (v >> 3) * 512 + (v & 7) * 64, 0);
          doff[r] = XA_OFF + rank * 65536 + v * 1024;
        }
      };
      ld(0); ld(1);
      if (!get_call()) return;
      const unsigned long long t_start = __builtin_amdgcn_s_memrealtime();
      const int rot = v % 3;
      __amdgpu_buffer_rsrc_t dr[3]; unsigned pofs[3];
#pragma unroll
      for (int k = 0; k < 3; ++k) {
        int kk = k + rot; if (kk >= 3) kk -= 3;
        const int qi = (ri + 1 + kk) & 3;
        dr[k] = mk_rsrc((char*)p.ws[grp * 4 + qi] + slot * SLOT_BYTES, USED_BYTES);
        pofs[k] = XS_OFF + (2 - kk) * XMAJ_BYTES;
      }
#pragma unroll
      for (int s = 0; s < NRS * 3; ++s) {
        if (s >= WINS) {
          int n = WINS - 1;
          for (int t = s - WINS; t < s; ++t) if (t % 3 == 2 && t / 3 + 2 < NRS) ++n;
          wait_vm_n(n);
        }
        if (s % 3 == 0) {
          if (PACE > 0) pace(t_start, (s / 3) * PACE + stag);
          d[s / 3] = nz4(d[s / 3]);
        }
        st128<17>(d[s / 3], dr[s % 3], lane * 16, doff[s / 3] + (s / 3 < GMAJ ? pofs[s % 3] : 0u));
        if (s % 3 == 2 && s / 3 + 2 < NRS) ld(s / 3 + 2);
      }
      if (lane == 0) __hip_atomic_store(cntp, call, __ATOMIC_RELAXED, __HIP_MEMORY_SCOPE_AGENT);
      return;
    } else {
      auto wr = mk_rsrc(p.w, KK * NL * 2);
      BRaw dw[4]; u32x4 dx[GMIN + 1]; unsigned doff[GMIN + 1];
      const int oi = v & 3;
      const unsigned lw = ((lane >> 4) * 8) * 512 + oi * 128 + (lane & 15) * 8;
      const int rot = v & 3;
      auto ld = [&](int r) {
        if (r < 4) {
          const int blk = r * 16 + (v >> 2);
#pragma unroll
          for (int jj = 0; jj < 8; ++jj) dw[r].r[jj] = __builtin_amdgcn_raw_buffer_load_b64(wr, lw + jj * 512, blk * 32 * 512, 0);
        } else if (r < 9) {
          const int e = (r - 4) * 64 + v, c = e / 40, rem = e % 40;
          const int g = rem >> 3, w = rem & 7;
          dx[r - 4] = __builtin_amdgcn_raw_buffer_load_b128(xr, lx, (ROW_MIN + g * 16) * 4096 + c * 512 + w * 64, 0);
          doff[r - 4] = XO_OFF + ri * XMIN_BYTES + ((c * GMIN + g) * 8 + w) * 1024;
        } else {
          dx[5] = __builtin_amdgcn_raw_buffer_load_b128(xr, lx, ROW_ALL * 4096 + (v >> 3) * 512 + (v & 7) * 64, 0);
          doff[5] = XA_OFF + rank * 65536 + v * 1024;
        }
      };
      ld(0); ld(1);
      if (!get_call()) return;
      const unsigned long long t_start = __builtin_amdgcn_s_memrealtime();
      __amdgpu_buffer_rsrc_t dr[4];
#pragma unroll
      for (int k = 0; k < 4; ++k) dr[k] = mk_rsrc((char*)p.ws[(grp ^ 1) * 4 + ((k + rot) & 3)] + slot * SLOT_BYTES, USED_BYTES);
      bf16x8 wf[4];
      auto step = [&](int s) {
        if (s >= WINC) {
          int n = WINC - 1;
          for (int t = s - WINC; t < s; ++t) if (t % 4 == 3 && t / 4 + 2 < 10) n += (t / 4 + 2 < 4) ? 8 : 1;
          wait_vm_n(n);
        }
        if (s % 4 == 0 && PACE > 0) pace(t_start, (s / 4) * PACE + stag);
        if (s < 16) {
          if (s % 4 == 0) perm_b(wf, dw[s / 4]);
          st128<17>(nz4(__builtin_bit_cast(u32x4, wf[s % 4])), dr[0], lane * 16,
                    WS_OFF + ri * 262144 + (((s / 4) * 16 + (v >> 2)) * 4 + (s % 4)) * 1024);
        } else {
          if (s % 4 == 0) dx[(s - 16) / 4] = nz4(dx[(s - 16) / 4]);
          st128<17>(dx[(s - 16) / 4], dr[s % 4], lane * 16, doff[(s - 16) / 4]);
        }
        if (s % 4 == 3 && s / 4 + 2 < 10) ld(s / 4 + 2);
      };
#pragma unroll
      for (int s = 0; s < 16; ++s) step(s);
#pragma unroll
      for (int s = 16; s < 40; ++s) step(s);
      Waiter wt{tmo, call, 0u, 0ull, false};
      collect<4, 2>(p, myws + slot * SLOT_BYTES, v & 3, 64 + (v >> 2), 16, grp, lane, wt);
      if (lane == 0) __hip_atomic_store(cntp, call, __ATOMIC_RELAXED, __HIP_MEMORY_SCOPE_AGENT);
      return;
    }
  }

  const int pi = tab & 63, ct = (tab >> 6) & 7, extg = (int)((tab >> 9) & 15) - 1, zi = (int)((tab >> 13) & 127) - 1;
  const int l15 = lane & 15, q = lane >> 4;
  const bool ext = extg >= 0, wremote = ct >= 4;
  const unsigned cv = __hip_atomic_load(cntp, __ATOMIC_RELAXED, __HIP_MEMORY_SCOPE_AGENT);
  const int owner = (grp ^ 1) * 4 + (ct & 3);
  const unsigned* credp = (const unsigned*)(myws + CRED_OFF) + (wremote ? owner : rank) * 32;
  unsigned cred = __hip_atomic_load(credp, __ATOMIC_RELAXED, __HIP_MEMORY_SCOPE_SYSTEM);
  bf16x8 bf[NCH][4];
  unsigned call, slot;
  Waiter wt{tmo, 0u, 0u, 0ull, false};
  if (!wremote) {
    auto wr = mk_rsrc((const char*)p.w + ct * 128, KK * NL * 2 - ct * 128);
    const unsigned wv = (wave * 32 + q * 8) * 512 + l15 * 8;
    BRaw br[NCH];
#pragma unroll
    for (int c = 0; c < NCH; ++c)
#pragma unroll
      for (int jj = 0; jj < 8; ++jj) br[c].r[jj] = __builtin_amdgcn_raw_buffer_load_b64(wr, wv + jj * 512, c * KC * 512, 0);
    asm volatile("" ::: "memory");
#pragma unroll
    for (int c = 0; c < NCH; ++c) {
      perm_b(bf[c], br[c]);
#pragma unroll
      for (int i = 0; i < 4; ++i) asm volatile("" : "+v"(bf[c][i]));
    }
    call = __builtin_amdgcn_readfirstlane(cv) + 1u;
    slot = call & 1u;
    wt.call = call;
  } else {
    call = __builtin_amdgcn_readfirstlane(cv) + 1u;
    slot = call & 1u;
    wt.call = call;
    auto wr = mk_rsrc(myws + slot * SLOT_BYTES + WS_OFF + (ct & 3) * 262144, 262144);
    const unsigned wv = wave * 4096 + lane * 16;
#pragma unroll
    for (int k = 0; k < NCH / 2; ++k) {
      u32x4 f[2][4];
      wt.reset();
      while (true) {
        asm volatile("" ::: "memory");
        f[0][3] = __builtin_amdgcn_raw_buffer_load_b128(wr, wv + 3072, (2 * k) * 32768, 16);
        f[1][3] = __builtin_amdgcn_raw_buffer_load_b128(wr, wv + 3072, (2 * k + 1) * 32768, 16);
        if (__all(umin(umin4(f[0][3]), umin4(f[1][3])) != 0u) || wt.dead) break;
        wt.tick(lane);
        __builtin_amdgcn_s_sleep(8);
      }
      while (true) {
        asm volatile("" ::: "memory");
#pragma unroll
        for (int h = 0; h < 2; ++h)
#pragma unroll
          for (int i = 0; i < 3; ++i) f[h][i] = __builtin_amdgcn_raw_buffer_load_b128(wr, wv + i * 1024, (2 * k + h) * 32768, 16);
        if (__all(umin(umin(umin4(f[0][0]), umin(umin4(f[0][1]), umin4(f[0][2]))), umin(umin4(f[1][0]), umin(umin4(f[1][1]), umin4(f[1][2])))) != 0u) || wt.dead) break;
        wt.tick(lane);
      }
#pragma unroll
      for (int h = 0; h < 2; ++h)
#pragma unroll
        for (int i = 0; i < 4; ++i) { bf[2 * k + h][i] = __builtin_bit_cast(bf16x8, f[h][i]); asm volatile("" : "+v"(bf[2 * k + h][i])); }
    }
  }
  char* slotp = myws + slot * SLOT_BYTES;
  if (zi >= 0) {
    auto zr = mk_rsrc(myws + (slot ^ 1u) * SLOT_BYTES, USED_BYTES);
    const u32x4 zero = {0u, 0u, 0u, 0u};
    const unsigned z0 = (unsigned)(zi * 8 + wave) * ZPW;
#pragma unroll
    for (int i = 0; i < ZPW; ++i) st128<17>(zero, zr, lane * 16, (z0 + i) * 1024u);
    asm volatile("s_waitcnt vmcnt(0)" ::: "memory");
    __syncthreads();
    if (wave == 0) {
      unsigned t = 0;
      if (lane == 0) t = __hip_atomic_fetch_add((unsigned*)(myws + TICK_OFF), 1u, __ATOMIC_RELAXED, __HIP_MEMORY_SCOPE_AGENT);
      t = __builtin_amdgcn_readfirstlane(t);
      if (t % NZB == NZB - 1 && lane < R - 1) {
        const int peer = (rank + 1 + lane) & 7;
        char* pw = (char*)p.ws[0];
#pragma unroll
        for (int i = 1; i < R; ++i) pw = (peer == i) ? (char*)p.ws[i] : pw;
        __hip_atomic_store((unsigned*)(pw + CRED_OFF) + rank * 32, call, __ATOMIC_RELAXED, __HIP_MEMORY_SCOPE_SYSTEM);
      }
    }
  }
  if (ext) {
    auto xr = mk_rsrc(p.x, ML * KK * 2);
    const unsigned xl_off = (extg * 16 + l15) * 4096 + (wave * 32 + q * 8) * 2;
    f32x4 accl[4];
#pragma unroll
    for (int jj = 0; jj < 4; ++jj) accl[jj] = (f32x4){0.f, 0.f, 0.f, 0.f};
#pragma unroll
    for (int c = 0; c < NCH; ++c) {
      const bf16x8 xf = __builtin_bit_cast(bf16x8, __builtin_amdgcn_raw_buffer_load_b128(xr, xl_off, c * (KC * 2), 0));
#pragma unroll
      for (int i = 0; i < 4; ++i) accl[i] = __builtin_amdgcn_mfma_f32_16x16x32_bf16(bf[c][i], xf, accl[i], 0, 0, 0);
    }
#pragma unroll
    for (int r = 0; r < 4; ++r) {
      f32x4 vv = {accl[0][r], accl[1][r], accl[2][r], accl[3][r]};
      *(f32x4*)&red[(wave * 16 + l15) * RED_LD + q * 16 + r * 4] = vv;
    }
    __syncthreads();
    if (threadIdx.x < 256) {
      const int t = threadIdx.x, row = t >> 4, cg = t & 15;
      f32x4 s = {0.f, 0.f, 0.f, 0.f};
#pragma unroll
      for (int ww = 0; ww < 8; ++ww) s += *(const f32x4*)&red[(ww * 16 + row) * RED_LD + cg * 4];
      u32x2 o;
      o.x = pack_bf16(silu(s[0]), silu(s[1]));
      o.y = pack_bf16(silu(s[2]), silu(s[3]));
      char* op = (char*)p.out + (size_t)(rank * ML + extg * 16 + row) * (NL * 2) + ct * 128 + cg * 8;
      *(u32x2*)op = o;
    }
    __syncthreads();
  }
  __amdgpu_buffer_rsrc_t ar[2]; unsigned a_off[2], acs[2]; bool chk[2];
#pragma unroll
  for (int mh = 0; mh < 2; ++mh) {
    int same, sidx, sg;
    decode_pair(pi, mh, same, sidx, sg);
    const bool alocal = (same == 1 && sidx == ri) || (same == 2 && sidx == rank);
    chk[mh] = !alocal;
    acs[mh] = alocal ? 512u : (same == 1 ? GMAJ * 8192u : (same == 0 ? GMIN * 8192u : 8192u));
    const unsigned abase = same == 1 ? XS_OFF + ((sidx - ri + 3) & 3) * XMAJ_BYTES : (same == 0 ? XO_OFF + sidx * XMIN_BYTES : XA_OFF + sidx * 65536u);
    ar[mh] = alocal ? mk_rsrc(p.x, ML * KK * 2) : mk_rsrc(slotp + abase, XMAJ_BYTES);
    a_off[mh] = alocal ? ((same == 2 ? ROW_ALL : sg * 16) + l15) * 4096u + (wave * 32 + q * 8) * 2 : (sg * 8 + wave) * 1024u + lane * 16;
  }
  u32x4 a[NCH][2];
  {
    const unsigned tw = pi < 8 ? 0u : (pi < 16 ? T_B : (pi < 26 ? T_MIN : (pi < 30 ? T_C : T_ALL)));
    while ((unsigned)(__builtin_amdgcn_s_memrealtime() - t_entry0) < tw) __builtin_amdgcn_s_sleep(8);
  }
#pragma unroll
  for (int mh = 0; mh < 2; ++mh) a[0][mh] = __builtin_amdgcn_raw_buffer_load_b128(ar[mh], a_off[mh], 0, 16);
  f32x4 acc[2][4];
#pragma unroll
  for (int mh = 0; mh < 2; ++mh)
#pragma unroll
    for (int jj = 0; jj < 4; ++jj) acc[mh][jj] = (f32x4){0.f, 0.f, 0.f, 0.f};
  unsigned tmov = 0;
#pragma unroll
  for (int c = 0; c < NCH; ++c) {
    wt.reset();
    while (true) {
      const unsigned m0 = chk[0] ? umin4(a[c][0]) : 1u, m1 = chk[1] ? umin4(a[c][1]) : 1u;
      if (__all(umin(m0, m1) != 0u) || wt.dead) break;
      wt.tick(lane);
      if (c == 0) __builtin_amdgcn_s_sleep(FIRST_SLEEP); else __builtin_amdgcn_s_sleep(1);
      asm volatile("" ::: "memory");
#pragma unroll
      for (int n = c; n < (c == 0 ? 1 : c + APF) && n < NCH; ++n)
#pragma unroll
        for (int mh = 0; mh < 2; ++mh) a[n][mh] = __builtin_amdgcn_raw_buffer_load_b128(ar[mh], a_off[mh], n * acs[mh], 16);
    }
    if (c == 0) {
#pragma unroll
      for (int n = 1; n < NCH; ++n)
#pragma unroll
        for (int mh = 0; mh < 2; ++mh) a[n][mh] = __builtin_amdgcn_raw_buffer_load_b128(ar[mh], a_off[mh], n * acs[mh], 16);
    }
    if (c == NCH - 1) tmov = __hip_atomic_load(tmo, __ATOMIC_RELAXED, __HIP_MEMORY_SCOPE_AGENT);
#pragma unroll
    for (int mh = 0; mh < 2; ++mh) {
      const bf16x8 af = __builtin_bit_cast(bf16x8, a[c][mh]);
#pragma unroll
      for (int i = 0; i < 4; ++i) acc[mh][i] = __builtin_amdgcn_mfma_f32_16x16x32_bf16(bf[c][i], af, acc[mh][i], 0, 0, 0);
    }
  }
  if (wt.dead) {
#pragma unroll
    for (int mh = 0; mh < 2; ++mh)
#pragma unroll
      for (int jj = 0; jj < 4; ++jj) acc[mh][jj] = (f32x4){__builtin_nanf(""), __builtin_nanf(""), __builtin_nanf(""), __builtin_nanf("")};
  }
#pragma unroll
  for (int mh = 0; mh < 2; ++mh)
#pragma unroll
    for (int r = 0; r < 4; ++r) {
      f32x4 vv = {acc[mh][0][r], acc[mh][1][r], acc[mh][2][r], acc[mh][3][r]};
      *(f32x4*)&red[(wave * 32 + mh * 16 + l15) * RED_LD + q * 16 + r * 4] = vv;
    }
  __syncthreads();
  if (!wremote) {
    const int t = threadIdx.x;
    const int row = t >> 4, cg = t & 15;
    f32x4 s0 = {0.f, 0.f, 0.f, 0.f};
#pragma unroll
    for (int ww = 0; ww < 8; ++ww) s0 += *(const f32x4*)&red[(ww * 32 + row) * RED_LD + cg * 4];
    u32x2 o;
    o.x = pack_bf16(silu(s0[0]), silu(s0[1]));
    o.y = pack_bf16(silu(s0[2]), silu(s0[3]));
    if (__builtin_amdgcn_readfirstlane(tmov) == call) o = (u32x2){0x7fc07fc0u, 0x7fc07fc0u};
    int s2, i2, g2;
    decode_pair(pi, wave >> 2, s2, i2, g2);
    const int grow = global_row(grp, s2, i2, g2) + (wave & 3) * 4;
    auto orr = mk_rsrc(p.out, R * ML * NL * 2);
    __builtin_amdgcn_raw_buffer_store_b64(o, orr, (lane >> 4) * (NL * 2) + (lane & 15) * 8, grow * (NL * 2) + ct * 128, 0);
  } else if (wave < 4) {
    const int t = threadIdx.x;
    const int row = t >> 3, cg = t & 7;
    f32x4 s0 = {0.f, 0.f, 0.f, 0.f}, s1 = {0.f, 0.f, 0.f, 0.f};
#pragma unroll
    for (int ww = 0; ww < 8; ++ww) {
      s0 += *(const f32x4*)&red[(ww * 32 + row) * RED_LD + cg * 8];
      s1 += *(const f32x4*)&red[(ww * 32 + row) * RED_LD + cg * 8 + 4];
    }
    u32x4 o;
    o.x = pack_bf16(silu(s0[0]), silu(s0[1]));
    o.y = pack_bf16(silu(s0[2]), silu(s0[3]));
    o.z = pack_bf16(silu(s1[0]), silu(s1[1]));
    o.w = pack_bf16(silu(s1[2]), silu(s1[3]));
    bool bad = __builtin_amdgcn_readfirstlane(tmov) == call;
    if (wremote) {
      if (!__all((int)(cred - (call - 1u)) >= 0)) {
        wt.reset();
        while (true) {
          cred = __hip_atomic_load(credp, __ATOMIC_RELAXED, __HIP_MEMORY_SCOPE_SYSTEM);
          if (__all((int)(cred - (call - 1u)) >= 0)) break;
          wt.tick(lane);
          if (wt.dead) break;
          __builtin_amdgcn_s_sleep(2);
        }
      }
      if (bad || wt.dead) o = (u32x4){0x7fc07fc0u, 0x7fc07fc0u, 0x7fc07fc0u, 0x7fc07fc0u};
      auto osr = mk_rsrc((char*)p.ws[owner] + slot * SLOT_BYTES + OS_OFF + ri * 131072, 131072);
      st128<17>(nz4(o), osr, lane * 16, pi * 4096 + wave * 1024);
    } else {
      if (bad) o = (u32x4){0x7fc07fc0u, 0x7fc07fc0u, 0x7fc07fc0u, 0x7fc07fc0u};
      int s2, i2, g2;
      decode_pair(pi, wave >> 1, s2, i2, g2);
      const int grow = global_row(grp, s2, i2, g2) + (wave & 1) * 8;
      auto orr = mk_rsrc(p.out, R * ML * NL * 2);
      st128<0>(o, orr, (lane >> 3) * (NL * 2) + (lane & 7) * 16, grow * (NL * 2) + ct * 128);
    }
  }
  if (threadIdx.x == 0) __hip_atomic_store(cntp, call, __ATOMIC_RELAXED, __HIP_MEMORY_SCOPE_AGENT);
}

}

extern "C" void kernel_launch(int rank, int nranks, ncclComm_t comm, hipStream_t stream,
                              void* const* d_in, const int* in_sizes, int n_in,
                              void* d_out, int out_size, const int* in_dtypes, int out_dtype,
                              void* const* d_out_all, void* const* d_ws_all, size_t ws_bytes) {
  if (nranks != R || (unsigned)rank >= (unsigned)R || ws_bytes < CTRL_OFF + (1u << 20) || n_in < 2 ||
      in_sizes[0] != ML * KK || in_sizes[1] != KK * NL || out_size != R * ML * NL) {
    fprintf(stderr, "kernel_launch: unexpected shapes\n");
    return;
  }
  Params p;
  p.x = d_in[0]; p.w = d_in[1]; p.out = d_out; p.rank = rank; p.pad = 0;
  for (int i = 0; i < R; ++i) p.ws[i] = d_ws_all[i];
  const int ri = rank & 3;
  unsigned tiles[256]; int nt = 0;
  int nextc[4] = {0, 0, 0, 0}, nzb = 0;
  for (int pi = 0; pi < 34; ++pi) {
    const bool localpair = (pi < 16 && (pi & 3) == ri) || (pi == 26 + ri);
    for (int ct = 0; ct < (pi < 30 ? 8 : 4); ++ct) {
      if (localpair && ct < 4) continue;
      unsigned e = (unsigned)pi | ((unsigned)ct << 6);
      if (ct < 4 && pi >= 16 && nextc[ct] < 10) e |= (unsigned)(++nextc[ct]) << 9;
      if (pi >= 8 && ct < 4 && nzb < NZB) e |= (unsigned)(++nzb) << 13;
      tiles[nt++] = e;
    }
  }
  for (int b = 0; b < 256; ++b) p.tab[b] = (unsigned)K_IDLE << 20;
  for (int b = 0; b < NPUSH; ++b) p.tab[b] = (unsigned)K_PUSH << 20;
  for (int b = NPUSH; b < NPUSH + 4; ++b) p.tab[b] = (unsigned)K_COLL << 20;
  {
    int t = 0;
    for (int l = 0; l < 8; ++l)
      for (int b = NPUSH + 4; b < 256; ++b)
        if ((b & 7) == l && t < nt) p.tab[b] = tiles[t++];
  }
  ag_gemm_kernel<<<256, 512, 0, stream>>>(p);
}
```

```cpp
#include <hip/hip_runtime.h>
#include <rccl/rccl.h>
#include <cstdio>
#include <cstdint>

namespace {

constexpr int R = 8, ML = 256, KK = 2048, NL = 256;
constexpr int NCH = 8, KC = KK / NCH;
constexpr int GMAJ = 11, GMIN = 4;
constexpr int ROW_MIN = GMAJ * 16, ROW_ALL = (GMAJ + GMIN) * 16;
constexpr unsigned XMAJ_BYTES = GMAJ * 65536;
constexpr unsigned XMIN_BYTES = GMIN * 65536;
constexpr unsigned XS_OFF = 0;
constexpr unsigned XO_OFF = 3 * XMAJ_BYTES;
constexpr unsigned XA_OFF = XO_OFF + 4 * XMIN_BYTES;
constexpr unsigned WS_OFF = XA_OFF + 8 * 65536;
constexpr unsigned OS_OFF = WS_OFF + 4 * 262144;
constexpr unsigned USED_BYTES = OS_OFF + 4 * 131072;
constexpr unsigned SLOT_BYTES = 6u << 20;
constexpr unsigned CTRL_OFF = 16u << 20;
constexpr unsigned CNT_OFF = CTRL_OFF;
constexpr unsigned TMO_OFF = CNT_OFF + 256 * 128;
constexpr unsigned TICK_OFF = TMO_OFF + 128;
constexpr unsigned CRED_OFF = TICK_OFF + 128;
constexpr unsigned WS_NEED = CRED_OFF + R * 128;
constexpr unsigned long long DEADLINE = 50000000ull;
constexpr int NPUSH = 16, NCOLL = 4;
constexpr int NZB = 81;
constexpr int ZPW = 8;

static_assert(NZB * 8 * ZPW * 1024 == USED_BYTES, "the zeroing blocks cover the slot exactly");
static_assert(2 * SLOT_BYTES <= CTRL_OFF && USED_BYTES <= SLOT_BYTES, "workspace layout");
constexpr int WINS = 4, WINC = 5;
constexpr unsigned PACE = 115, PACEC = 120;
constexpr unsigned T_W = 300;
constexpr unsigned T_B = 450, T_MIN = 550, T_C = 900, T_ALL = 1000;
constexpr int APF = 3;

typedef unsigned u32x4 __attribute__((ext_vector_type(4)));
typedef unsigned u32x2 __attribute__((ext_vector_type(2)));
typedef float f32x4 __attribute__((ext_vector_type(4)));
typedef short bf16x8 __attribute__((ext_vector_type(8)));

struct Params {
  const void* x; const void* w; void* out;
  char* myws;
  char* sg[3];
  char* og[4];
  int rank; int pad;
  unsigned tab[256];
};

__device__ __forceinline__ __amdgpu_buffer_rsrc_t mk_rsrc(const void* p, unsigned bytes) {
  return __builtin_amdgcn_make_buffer_rsrc(const_cast<void*>(p), 0, bytes, 0x00020000);
}
template <int AUX>
__device__ __forceinline__ void st128(u32x4 v, __amdgpu_buffer_rsrc_t r, unsigned voff, unsigned soff) {
  __builtin_amdgcn_raw_buffer_store_b128(v, r, voff, soff, AUX);
  asm volatile("s_nop 1" ::"v"(v));
}
__device__ __forceinline__ void wait_vm_n(int n) {
  switch (n) {
#define C(N) case N: asm volatile("s_waitcnt vmcnt(" #N ")" ::: "memory"); break;
    C(0) C(1) C(2) C(3) C(4) C(5) C(6) C(7) C(8) C(9) C(10) C(11) C(12) C(13) C(14) C(15) C(16) C(17) C(18) C(19) C(20) C(21) C(22) C(23) C(24)
#undef C
    default: asm volatile("s_waitcnt vmcnt(0)" ::: "memory"); break;
  }
}
__device__ __forceinline__ unsigned pack_bf16(float lo, float hi) {
  unsigned r; asm("v_cvt_pk_bf16_f32 %0, %1, %2" : "=v"(r) : "v"(lo), "v"(hi)); return r;
}
__device__ __forceinline__ float silu(float y) {
  float t = fminf(fmaxf(y, -60.f), 60.f);
  float e = __builtin_amdgcn_exp2f(t * -1.44269504088896f);
  return y * __builtin_amdgcn_rcpf(1.f + e);
}
__device__ __forceinline__ unsigned nz(unsigned v) { return v == 0u ? 0x80000000u : v; }
__device__ __forceinline__ u32x4 nz4(u32x4 v) { v.x = nz(v.x); v.y = nz(v.y); v.z = nz(v.z); v.w = nz(v.w); return v; }
__device__ __forceinline__ unsigned umin(unsigned a, unsigned b) { return a < b ? a : b; }
__device__ __forceinline__ unsigned umin4(u32x4 v) { return umin(umin(v.x, v.y), umin(v.z, v.w)); }

__device__ __forceinline__ void decode_pair(int pi, int mh, int& same, int& idx, int& g) {
  if (pi < 16) { same = 1; idx = pi & 3; g = 2 * (pi >> 2) + mh; }
  else if (pi < 24) { same = 0; idx = pi & 3; g = 2 * ((pi - 16) >> 2) + mh; }
  else if (pi < 28) { same = 1; idx = pi - 24; g = 8 + mh; }
  else if (pi < 30) { same = 1; idx = 2 * (pi - 28) + mh; g = 10; }
  else { same = 2; idx = 2 * (pi - 30) + mh; g = 0; }
}
__device__ __forceinline__ int global_row(int grp, int same, int idx, int g) {
  return same == 2 ? idx * ML + ROW_ALL : (same ? (grp * 4 + idx) * ML + 16 * g : ((grp ^ 1) * 4 + idx) * ML + ROW_MIN + 16 * g);
}

__device__ __forceinline__ char* pick4(char* const (&t)[4], int i) { return i == 0 ? t[0] : (i == 1 ? t[1] : (i == 2 ? t[2] : t[3])); }
__device__ __forceinline__ void pace(unsigned long long t_start, unsigned ticks) {
  while ((unsigned)(__builtin_amdgcn_s_memrealtime() - t_start) < ticks) __builtin_amdgcn_s_sleep(1);
}
struct Waiter {
  unsigned* tmo; unsigned call; unsigned it; unsigned long long t0; bool dead;
  __device__ __forceinline__ void tick(int lane) {
    if ((++it & 63u) == 0u) {
      const unsigned long long now = __builtin_amdgcn_s_memrealtime();
      if (t0 == 0ull) t0 = now;
      if (__builtin_amdgcn_readfirstlane(__hip_atomic_load(tmo, __ATOMIC_RELAXED, __HIP_MEMORY_SCOPE_AGENT)) == call || now - t0 > DEADLINE) {
        if (lane == 0) __hip_atomic_store(tmo, call, __ATOMIC_RELAXED, __HIP_MEMORY_SCOPE_AGENT);
        dead = true;
      }
    }
  }
  __device__ __forceinline__ void reset() { it = 0; t0 = 0ull; }
};

struct BRaw { u32x2 r[8]; };
__device__ __forceinline__ void perm_b(bf16x8 (&f)[4], const BRaw& b) {
  u32x4 f0, f1, f2, f3;
#pragma unroll
  for (int pp = 0; pp < 4; ++pp) {
    f0[pp] = __builtin_amdgcn_perm(b.r[2 * pp + 1].x, b.r[2 * pp].x, 0x05040100u);
    f1[pp] = __builtin_amdgcn_perm(b.r[2 * pp + 1].x, b.r[2 * pp].x, 0x07060302u);
    f2[pp] = __builtin_amdgcn_perm(b.r[2 * pp + 1].y, b.r[2 * pp].y, 0x05040100u);
    f3[pp] = __builtin_amdgcn_perm(b.r[2 * pp + 1].y, b.r[2 * pp].y, 0x07060302u);
  }
  f[0] = __builtin_bit_cast(bf16x8, f0); f[1] = __builtin_bit_cast(bf16x8, f1);
  f[2] = __builtin_bit_cast(bf16x8, f2); f[3] = __builtin_bit_cast(bf16x8, f3);
}

constexpr int RED_LD = 68;
constexpr int RED_MAIN = 8 * 32 * RED_LD;

template <int NP, int NB>
__device__ __forceinline__ void collect(const Params& p, char* slotp, int hi, int pp0, int ppstep, int grp, int lane, Waiter& wt) {
  auto sr = mk_rsrc(slotp + OS_OFF + hi * 131072, 131072);
  auto orr = mk_rsrc(p.out, R * ML * NL * 2);
#pragma unroll 1
  for (int i0 = 0; i0 < NP; i0 += NB) {
    u32x4 v[NB]; unsigned tm;
    wt.reset();
    while (true) {
      asm volatile("" ::: "memory");
      unsigned m = 0xffffffffu;
#pragma unroll
      for (int b = 0; b < NB; ++b) {
        const int pp = pp0 + (i0 + b) * ppstep;
        v[b] = __builtin_amdgcn_raw_buffer_load_b128(sr, lane * 16, (pp < 120 ? pp : pp0) * 1024, 16);
      }
      tm = __hip_atomic_load(wt.tmo, __ATOMIC_RELAXED, __HIP_MEMORY_SCOPE_AGENT);
#pragma unroll
      for (int b = 0; b < NB; ++b) m = umin(m, umin4(v[b]));
      if (__all(m != 0u) || wt.dead) break;
      wt.tick(lane);
      __builtin_amdgcn_s_sleep(1);
    }
    const bool bad = wt.dead || __builtin_amdgcn_readfirstlane(tm) == wt.call;
#pragma unroll
    for (int b = 0; b < NB; ++b) {
      const int pp = pp0 + (i0 + b) * ppstep;
      if (pp < 120) {
        if (bad) v[b] = (u32x4){0x7fc07fc0u, 0x7fc07fc0u, 0x7fc07fc0u, 0x7fc07fc0u};
        int same, idx, g;
        decode_pair(pp >> 2, (pp >> 1) & 1, same, idx, g);
        const int row = global_row(grp ^ 1, same, idx, g) + (pp & 1) * 8;
        st128<0>(v[b], orr, (lane >> 3) * (NL * 2) + (lane & 7) * 16, row * (NL * 2) + hi * 128);
      }
    }
  }
}

__global__ __launch_bounds__(512, 2) void ag_gemm_kernel(Params p) {
  __shared__ __attribute__((aligned(16))) float red[RED_MAIN];
  const int b = blockIdx.x;
  const int lane = threadIdx.x & 63;
  const int wave = __builtin_amdgcn_readfirstlane(threadIdx.x >> 6);
  const int rank = p.rank, grp = rank >> 2, ri = rank & 3;
  char* myws = p.myws;
  unsigned* cntp = (unsigned*)(myws + CNT_OFF + b * 128);
  const unsigned long long t_entry0 = __builtin_amdgcn_s_memrealtime();
  unsigned* tmo = (unsigned*)(myws + TMO_OFF);

  if (b < NPUSH + NCOLL) {
    cntp += wave;
    if (b >= NPUSH) {
      const unsigned call = __builtin_amdgcn_readfirstlane(__hip_atomic_load(cntp, __ATOMIC_RELAXED, __HIP_MEMORY_SCOPE_AGENT)) + 1u;
      Waiter wt{tmo, call, 0u, 0ull, false};
      const int dv = (b - NPUSH) * 8 + wave;
      collect<8, 1>(p, myws + (call & 1u) * SLOT_BYTES, dv & 3, dv >> 2, 8, grp, lane, wt);
      if (lane == 0) __hip_atomic_store(cntp, call, __ATOMIC_RELAXED, __HIP_MEMORY_SCOPE_AGENT);
      return;
    }
    const int v = b * 4 + (wave & 3);
    const bool swave = wave < 4;
    auto xr = mk_rsrc(p.x, ML * KK * 2);
    const unsigned lx = (lane & 15) * 4096 + (lane >> 4) * 16;
    const unsigned cv = __hip_atomic_load(cntp, __ATOMIC_RELAXED, __HIP_MEMORY_SCOPE_AGENT);
    const unsigned* credp = (const unsigned*)(myws + CRED_OFF) + ((rank + 1 + (lane % 7)) & 7) * 32;
    unsigned cred = __hip_atomic_load(credp, __ATOMIC_RELAXED, __HIP_MEMORY_SCOPE_SYSTEM);
    const unsigned stag_s = (v * PACE) / 64, stag_c = (v * PACEC) / 64;
    unsigned call, slot;
    auto get_call = [&]() -> bool {
      call = __builtin_amdgcn_readfirstlane(cv) + 1u;
      slot = call & 1u;
      if (!__all((int)(cred - (call - 1u)) >= 0)) {
        Waiter w2{tmo, call, 0u, 0ull, false};
        while (true) {
          __builtin_amdgcn_s_sleep(2);
          cred = __hip_atomic_load(credp, __ATOMIC_RELAXED, __HIP_MEMORY_SCOPE_SYSTEM);
          if (__all((int)(cred - (call - 1u)) >= 0)) break;
          w2.tick(lane);
          if (w2.dead) return false;
        }
      }
      return true;
    };
    if (swave) {
      constexpr int NRS = GMAJ + 1;
      u32x4 d[NRS]; unsigned doff[NRS];
      auto ld = [&](int r) {
        if (r < GMAJ) {
          const int g0 = r < 4 ? 0 : (r < 8 ? 4 : 8), ng = r < 8 ? 4 : 3, rr = r - g0;
          const int e = rr * 64 + v, c = e / (8 * ng), rem = e % (8 * ng);
          const int g = g0 + (rem >> 3), w = rem & 7;
          d[r] = __builtin_amdgcn_raw_buffer_load_b128(xr, lx, g * 16 * 4096 + c * 512 + w * 64, 0);
          doff[r] = ((c * GMAJ + g) * 8 + w) * 1024;
        } else {
          d[r] = __builtin_amdgcn_raw_buffer_load_b128(xr, lx, ROW_ALL * 4096 + (v >> 3) * 512 + (v & 7) * 64, 0);
          doff[r] = XA_OFF + rank * 65536 + v * 1024;
        }
      };
      ld(0); ld(1);
      if (!get_call()) { if (lane == 0) __hip_atomic_store(cntp, call, __ATOMIC_RELAXED, __HIP_MEMORY_SCOPE_AGENT); return; }
      const unsigned long long t_start = __builtin_amdgcn_s_memrealtime();
      const int rot = v % 3;
      __amdgpu_buffer_rsrc_t dr[3]; unsigned pofs[3];
#pragma unroll
      for (int k = 0; k < 3; ++k) {
        int kk = k + rot; if (kk >= 3) kk -= 3;
        dr[k] = mk_rsrc((kk == 0 ? p.sg[0] : (kk == 1 ? p.sg[1] : p.sg[2])) + slot * SLOT_BYTES, USED_BYTES);
        pofs[k] = XS_OFF + (2 - kk) * XMAJ_BYTES;
      }
#pragma unroll
      for (int s = 0; s < NRS * 3; ++s) {
        if (s >= WINS) {
          int n = WINS - 1;
          for (int t = s - WINS; t < s; ++t) if (t % 3 == 2 && t / 3 + 2 < NRS) ++n;
          wait_vm_n(n);
        }
        if (s % 3 == 0) {
          pace(t_start, (s / 3) * PACE + stag_s);
          d[s / 3] = nz4(d[s / 3]);
        }
        st128<17>(d[s / 3], dr[s % 3], lane * 16, doff[s / 3] + (s / 3 < GMAJ ? pofs[s % 3] : 0u));
        if (s % 3 == 2 && s / 3 + 2 < NRS) ld(s / 3 + 2);
      }
      if (lane == 0) __hip_atomic_store(cntp, call, __ATOMIC_RELAXED, __HIP_MEMORY_SCOPE_AGENT);
      return;
    } else {
      auto wr = mk_rsrc(p.w, KK * NL * 2);
      constexpr int NRC = 4 + GMIN + 1;
      BRaw dw[4]; u32x4 dx[GMIN + 1]; unsigned doff[GMIN + 1];
      const int oi = v & 3;
      const unsigned lw = ((lane >> 4) * 8) * 512 + oi * 128 + (lane & 15) * 8;
      const int rot = v & 3;
      auto ld = [&](int r) {
        if (r < 4) {
          const int blk = r * 16 + (v >> 2);
#pragma unroll
          for (int jj = 0; jj < 8; ++jj) dw[r].r[jj] = __builtin_amdgcn_raw_buffer_load_b64(wr, lw + jj * 512, blk * 32 * 512, 0);
        } else if (r < 4 + GMIN) {
          const int e = (r - 4) * 64 + v, c = e / (8 * GMIN), rem = e % (8 * GMIN);
          const int g = rem >> 3, w = rem & 7;
          dx[r - 4] = __builtin_amdgcn_raw_buffer_load_b128(xr, lx, (ROW_MIN + g * 16) * 4096 + c * 512 + w * 64, 0);
          doff[r - 4] = XO_OFF + ri * XMIN_BYTES + ((c * GMIN + g) * 8 + w) * 1024;
        } else {
          dx[r - 4] = __builtin_amdgcn_raw_buffer_load_b128(xr, lx, ROW_ALL * 4096 + (v >> 3) * 512 + (v & 7) * 64, 0);
          doff[r - 4] = XA_OFF + rank * 65536 + v * 1024;
        }
      };
      ld(0); ld(1);
      const bool go = get_call();
      const unsigned long long t_start = __builtin_amdgcn_s_memrealtime();
      __amdgpu_buffer_rsrc_t dr[4];
#pragma unroll
      for (int k = 0; k < 4; ++k) dr[k] = mk_rsrc(pick4(p.og, (k + rot) & 3) + slot * SLOT_BYTES, USED_BYTES);
      bf16x8 wf[4];
      auto step = [&](int s) {
        if (s >= WINC) {
          int n = WINC - 1;
          for (int t = s - WINC; t < s; ++t) if (t % 4 == 3 && t / 4 + 2 < NRC) n += (t / 4 + 2 < 4) ? 8 : 1;
          wait_vm_n(n);
        }
        if (s % 4 == 0) pace(t_start, (s / 4) * PACEC + stag_c);
        if (s < 16) {
          if (s % 4 == 0) perm_b(wf, dw[s / 4]);
          st128<17>(nz4(__builtin_bit_cast(u32x4, wf[s % 4])), dr[0], lane * 16,
                    WS_OFF + ri * 262144 + (((s / 4) * 16 + (v >> 2)) * 4 + (s % 4)) * 1024);
        } else {
          if (s % 4 == 0) dx[(s - 16) / 4] = nz4(dx[(s - 16) / 4]);
          st128<17>(dx[(s - 16) / 4], dr[s % 4], lane * 16, doff[(s - 16) / 4]);
        }
        if (s % 4 == 3 && s / 4 + 2 < NRC) ld(s / 4 + 2);
      };
      if (go) {
#pragma unroll
        for (int s = 0; s < 16; ++s) step(s);
#pragma unroll
        for (int s = 16; s < 4 * NRC; ++s) step(s);
      }
      Waiter wt{tmo, call, 0u, 0ull, !go};
      collect<4, 2>(p, myws + slot * SLOT_BYTES, v & 3, 64 + (v >> 2), 16, grp, lane, wt);
      if (lane == 0) __hip_atomic_store(cntp, call, __ATOMIC_RELAXED, __HIP_MEMORY_SCOPE_AGENT);
      return;
    }
  }

  const unsigned tab = p.tab[b];
  const int pi = tab & 63, ct = (tab >> 6) & 7, extg = (int)((tab >> 9) & 15) - 1, zi = (int)((tab >> 13) & 127) - 1;
  const int l15 = lane & 15, q = lane >> 4;
  const bool ext = extg >= 0, wremote = ct >= 4;
  const unsigned cv = __hip_atomic_load(cntp, __ATOMIC_RELAXED, __HIP_MEMORY_SCOPE_AGENT);
  const int owner = (grp ^ 1) * 4 + (ct & 3);
  const unsigned* credp = (const unsigned*)(myws + CRED_OFF) + (wremote ? owner : rank) * 32;
  unsigned cred = __hip_atomic_load(credp, __ATOMIC_RELAXED, __HIP_MEMORY_SCOPE_SYSTEM);
  bf16x8 bf[NCH][4];
  unsigned call, slot;
  Waiter wt{tmo, 0u, 0u, 0ull, false};
  if (!wremote) {
    auto wr = mk_rsrc((const char*)p.w + ct * 128, KK * NL * 2 - ct * 128);
    const unsigned wv = (wave * 32 + q * 8) * 512 + l15 * 8;
    BRaw br[NCH];
#pragma unroll
    for (int c = 0; c < NCH; ++c)
#pragma unroll
      for (int jj = 0; jj < 8; ++jj) br[c].r[jj] = __builtin_amdgcn_raw_buffer_load_b64(wr, wv + jj * 512, c * KC * 512, 0);
    asm volatile("" ::: "memory");
#pragma unroll
    for (int c = 0; c < NCH; ++c) {
      perm_b(bf[c], br[c]);
#pragma unroll
      for (int i = 0; i < 4; ++i) asm volatile("" : "+v"(bf[c][i]));
    }
    call = __builtin_amdgcn_readfirstlane(cv) + 1u;
    slot = call & 1u;
    wt.call = call;
  } else {
    call = __builtin_amdgcn_readfirstlane(cv) + 1u;
    slot = call & 1u;
    wt.call = call;
    auto wr = mk_rsrc(myws + slot * SLOT_BYTES + WS_OFF + (ct & 3) * 262144, 262144);
    const unsigned wv = wave * 4096 + lane * 16;
    u32x4 f[NCH][4];
#pragma unroll
    for (int k = 0; k < NCH / 2; ++k) {
      while ((unsigned)(__builtin_amdgcn_s_memrealtime() - t_entry0) < T_W + k * PACEC) __builtin_amdgcn_s_sleep(4);
      wt.reset();
      while (true) {
        asm volatile("" ::: "memory");
        f[2 * k][3] = __builtin_amdgcn_raw_buffer_load_b128(wr, wv + 3072, (2 * k) * 32768, 16);
        f[2 * k + 1][3] = __builtin_amdgcn_raw_buffer_load_b128(wr, wv + 3072, (2 * k + 1) * 32768, 16);
        if (__all(umin(umin4(f[2 * k][3]), umin4(f[2 * k + 1][3])) != 0u) || wt.dead) break;
        wt.tick(lane);
        __builtin_amdgcn_s_sleep(8);
      }
#pragma unroll
      for (int h = 0; h < 2; ++h)
#pragma unroll
        for (int i = 0; i < 3; ++i) f[2 * k + h][i] = __builtin_amdgcn_raw_buffer_load_b128(wr, wv + i * 1024, (2 * k + h) * 32768, 16);
    }
#pragma unroll
    for (int c = 0; c < NCH; ++c) {
      wt.reset();
      while (!(__all(umin(umin4(f[c][0]), umin(umin4(f[c][1]), umin4(f[c][2]))) != 0u) || wt.dead)) {
        wt.tick(lane);
        asm volatile("" ::: "memory");
#pragma unroll
        for (int i = 0; i < 3; ++i) f[c][i] = __builtin_amdgcn_raw_buffer_load_b128(wr, wv + i * 1024, c * 32768, 16);
      }
#pragma unroll
      for (int i = 0; i < 4; ++i) { bf[c][i] = __builtin_bit_cast(bf16x8, f[c][i]); asm volatile("" : "+v"(bf[c][i])); }
    }
  }
  char* slotp = myws + slot * SLOT_BYTES;
  if (zi >= 0) {
    auto zr = mk_rsrc(myws + (slot ^ 1u) * SLOT_BYTES, USED_BYTES);
    const u32x4 zero = {0u, 0u, 0u, 0u};
    const unsigned z0 = (unsigned)(zi * 8 + wave) * ZPW;
#pragma unroll
    for (int i = 0; i < ZPW; ++i) st128<17>(zero, zr, lane * 16, (z0 + i) * 1024u);
    asm volatile("s_waitcnt vmcnt(0)" ::: "memory");
    __syncthreads();
    if (wave == 0) {
      unsigned t = 0;
      if (lane == 0) t = __hip_atomic_fetch_add((unsigned*)(myws + TICK_OFF), 1u, __ATOMIC_RELAXED, __HIP_MEMORY_SCOPE_AGENT);
      t = __builtin_amdgcn_readfirstlane(t);
      if (t % NZB == NZB - 1 && lane < R - 1) {
        char* pw = p.sg[0];
        pw = lane == 1 ? p.sg[1] : pw; pw = lane == 2 ? p.sg[2] : pw;
#pragma unroll
        for (int i = 0; i < 4; ++i) pw = lane == 3 + i ? p.og[i] : pw;
        __hip_atomic_store((unsigned*)(pw + CRED_OFF) + rank * 32, call, __ATOMIC_RELAXED, __HIP_MEMORY_SCOPE_SYSTEM);
      }
    }
  }
  if (ext) {
    auto xr = mk_rsrc(p.x, ML * KK * 2);
    const unsigned xl_off = (extg * 16 + l15) * 4096 + (wave * 32 + q * 8) * 2;
    f32x4 accl[4];
#pragma unroll
    for (int jj = 0; jj < 4; ++jj) accl[jj] = (f32x4){0.f, 0.f, 0.f, 0.f};
#pragma unroll
    for (int c = 0; c < NCH; ++c) {
      const bf16x8 xf = __builtin_bit_cast(bf16x8, __builtin_amdgcn_raw_buffer_load_b128(xr, xl_off, c * (KC * 2), 0));
#pragma unroll
      for (int i = 0; i < 4; ++i) accl[i] = __builtin_amdgcn_mfma_f32_16x16x32_bf16(bf[c][i], xf, accl[i], 0, 0, 0);
    }
#pragma unroll
    for (int r = 0; r < 4; ++r) {
      f32x4 vv = {accl[0][r], accl[1][r], accl[2][r], accl[3][r]};
      *(f32x4*)&red[(wave * 16 + l15) * RED_LD + q * 16 + r * 4] = vv;
    }
    __syncthreads();
    if (threadIdx.x < 256) {
      const int t = threadIdx.x, row = t >> 4, cg = t & 15;
      f32x4 s = {0.f, 0.f, 0.f, 0.f};
#pragma unroll
      for (int ww = 0; ww < 8; ++ww) s += *(const f32x4*)&red[(ww * 16 + row) * RED_LD + cg * 4];
      u32x2 o;
      o.x = pack_bf16(silu(s[0]), silu(s[1]));
      o.y = pack_bf16(silu(s[2]), silu(s[3]));
      char* op = (char*)p.out + (size_t)(rank * ML + extg * 16 + row) * (NL * 2) + ct * 128 + cg * 8;
      *(u32x2*)op = o;
    }
    __syncthreads();
  }
  __amdgpu_buffer_rsrc_t ar[2]; unsigned a_off[2], acs[2]; bool chk[2];
#pragma unroll
  for (int mh = 0; mh < 2; ++mh) {
    int same, sidx, sg;
    decode_pair(pi, mh, same, sidx, sg);
    const bool alocal = (same == 1 && sidx == ri) || (same == 2 && sidx == rank);
    chk[mh] = !alocal;
    acs[mh] = alocal ? 512u : (same == 1 ? GMAJ * 8192u : (same == 0 ? GMIN * 8192u : 8192u));
    const unsigned abase = same == 1 ? XS_OFF + ((sidx - ri + 3) & 3) * XMAJ_BYTES : (same == 0 ? XO_OFF + sidx * XMIN_BYTES : XA_OFF + sidx * 65536u);
    ar[mh] = alocal ? mk_rsrc(p.x, ML * KK * 2) : mk_rsrc(slotp + abase, XMAJ_BYTES);
    a_off[mh] = alocal ? ((same == 2 ? ROW_ALL : sg * 16) + l15) * 4096u + (wave * 32 + q * 8) * 2 : (sg * 8 + wave) * 1024u + lane * 16;
  }
  u32x4 a[NCH][2];
  {
    const unsigned tw = pi < 8 ? 0u : (pi < 16 ? T_B : (pi < 24 ? T_MIN : (pi < 30 ? T_C : T_ALL)));
    while ((unsigned)(__builtin_amdgcn_s_memrealtime() - t_entry0) < tw) __builtin_amdgcn_s_sleep(8);
  }
#pragma unroll
  for (int mh = 0; mh < 2; ++mh) a[0][mh] = __builtin_amdgcn_raw_buffer_load_b128(ar[mh], a_off[mh], 0, 16);
  f32x4 acc[2][4];
#pragma unroll
  for (int mh = 0; mh < 2; ++mh)
#pragma unroll
    for (int jj = 0; jj < 4; ++jj) acc[mh][jj] = (f32x4){0.f, 0.f, 0.f, 0.f};
  unsigned tmov = 0;
#pragma unroll
  for (int c = 0; c < NCH; ++c) {
    wt.reset();
    while (true) {
      const unsigned m0 = chk[0] ? umin4(a[c][0]) : 1u, m1 = chk[1] ? umin4(a[c][1]) : 1u;
      if (__all(umin(m0, m1) != 0u) || wt.dead) break;
      wt.tick(lane);
      if (c == 0) __builtin_amdgcn_s_sleep(2); else __builtin_amdgcn_s_sleep(1);
      asm volatile("" ::: "memory");
#pragma unroll
      for (int n = c; n < (c == 0 ? 1 : c + APF) && n < NCH; ++n)
#pragma unroll
        for (int mh = 0; mh < 2; ++mh) a[n][mh] = __builtin_amdgcn_raw_buffer_load_b128(ar[mh], a_off[mh], n * acs[mh], 16);
    }
    if (c == 0) {
#pragma unroll
      for (int n = 1; n < NCH; ++n)
#pragma unroll
        for (int mh = 0; mh < 2; ++mh) a[n][mh] = __builtin_amdgcn_raw_buffer_load_b128(ar[mh], a_off[mh], n * acs[mh], 16);
    }
    if (c == NCH - 1) tmov = __hip_atomic_load(tmo, __ATOMIC_RELAXED, __HIP_MEMORY_SCOPE_AGENT);
#pragma unroll
    for (int mh = 0; mh < 2; ++mh) {
      const bf16x8 af = __builtin_bit_cast(bf16x8, a[c][mh]);
#pragma unroll
      for (int i = 0; i < 4; ++i) acc[mh][i] = __builtin_amdgcn_mfma_f32_16x16x32_bf16(bf[c][i], af, acc[mh][i], 0, 0, 0);
    }
  }
  if (wt.dead) {
#pragma unroll
    for (int mh = 0; mh < 2; ++mh)
#pragma unroll
      for (int jj = 0; jj < 4; ++jj) acc[mh][jj] = (f32x4){__builtin_nanf(""), __builtin_nanf(""), __builtin_nanf(""), __builtin_nanf("")};
  }
#pragma unroll
  for (int mh = 0; mh < 2; ++mh)
#pragma unroll
    for (int r = 0; r < 4; ++r) {
      f32x4 vv = {acc[mh][0][r], acc[mh][1][r], acc[mh][2][r], acc[mh][3][r]};
      *(f32x4*)&red[(wave * 32 + mh * 16 + l15) * RED_LD + q * 16 + r * 4] = vv;
    }
  __syncthreads();
  if (!wremote) {
    const int t = threadIdx.x;
    const int row = t >> 4, cg = t & 15;
    f32x4 s0 = {0.f, 0.f, 0.f, 0.f};
#pragma unroll
    for (int ww = 0; ww < 8; ++ww) s0 += *(const f32x4*)&red[(ww * 32 + row) * RED_LD + cg * 4];
    u32x2 o;
    o.x = pack_bf16(silu(s0[0]), silu(s0[1]));
    o.y = pack_bf16(silu(s0[2]), silu(s0[3]));
    if (__builtin_amdgcn_readfirstlane(tmov) == call) o = (u32x2){0x7fc07fc0u, 0x7fc07fc0u};
    int s2, i2, g2;
    decode_pair(pi, wave >> 2, s2, i2, g2);
    const int grow = global_row(grp, s2, i2, g2) + (wave & 3) * 4;
    auto orr = mk_rsrc(p.out, R * ML * NL * 2);
    __builtin_amdgcn_raw_buffer_store_b64(o, orr, (lane >> 4) * (NL * 2) + (lane & 15) * 8, grow * (NL * 2) + ct * 128, 0);
  } else if (wave < 4) {
    const int t = threadIdx.x;
    const int row = t >> 3, cg = t & 7;
    f32x4 s0 = {0.f, 0.f, 0.f, 0.f}, s1 = {0.f, 0.f, 0.f, 0.f};
#pragma unroll
    for (int ww = 0; ww < 8; ++ww) {
      s0 += *(const f32x4*)&red[(ww * 32 + row) * RED_LD + cg * 8];
      s1 += *(const f32x4*)&red[(ww * 32 + row) * RED_LD + cg * 8 + 4];
    }
    u32x4 o;
    o.x = pack_bf16(silu(s0[0]), silu(s0[1]));
    o.y = pack_bf16(silu(s0[2]), silu(s0[3]));
    o.z = pack_bf16(silu(s1[0]), silu(s1[1]));
    o.w = pack_bf16(silu(s1[2]), silu(s1[3]));
    const bool bad = __builtin_amdgcn_readfirstlane(tmov) == call;
    if (!__all((int)(cred - (call - 1u)) >= 0)) {
      wt.reset();
      while (true) {
        cred = __hip_atomic_load(credp, __ATOMIC_RELAXED, __HIP_MEMORY_SCOPE_SYSTEM);
        if (__all((int)(cred - (call - 1u)) >= 0)) break;
        wt.tick(lane);
        if (wt.dead) break;
        __builtin_amdgcn_s_sleep(2);
      }
    }
    if (bad || wt.dead) o = (u32x4){0x7fc07fc0u, 0x7fc07fc0u, 0x7fc07fc0u, 0x7fc07fc0u};
    auto osr = mk_rsrc(pick4(p.og, ct & 3) + slot * SLOT_BYTES + OS_OFF + ri * 131072, 131072);
    st128<17>(nz4(o), osr, lane * 16, pi * 4096 + wave * 1024);
  }
  if (threadIdx.x == 0) __hip_atomic_store(cntp, call, __ATOMIC_RELAXED, __HIP_MEMORY_SCOPE_AGENT);
}

}

extern "C" void kernel_launch(int rank, int nranks, ncclComm_t comm, hipStream_t stream,
                              void* const* d_in, const int* in_sizes, int n_in,
                              void* d_out, int out_size, const int* in_dtypes, int out_dtype,
                              void* const* d_out_all, void* const* d_ws_all, size_t ws_bytes) {
  if (nranks != R || (unsigned)rank >= (unsigned)R || ws_bytes < WS_NEED || n_in < 2 ||
      in_sizes[0] != ML * KK || in_sizes[1] != KK * NL || out_size != R * ML * NL) {
    fprintf(stderr, "kernel_launch: unexpected shapes\n");
    return;
  }
  Params p;
  p.x = d_in[0]; p.w = d_in[1]; p.out = d_out; p.rank = rank; p.pad = 0;
  p.myws = (char*)d_ws_all[rank];
  for (int k = 0; k < 3; ++k) p.sg[k] = (char*)d_ws_all[(rank & 4) + (((rank & 3) + 1 + k) & 3)];
  for (int k = 0; k < 4; ++k) p.og[k] = (char*)d_ws_all[((rank & 4) ^ 4) + k];
  const int ri = rank & 3;
  unsigned tiles[256]; int nt = 0;
  int nextc[4] = {0, 0, 0, 0}, nzb = 0;
  for (int pi = 0; pi < 34; ++pi) {
    const bool localpair = (pi < 16 && (pi & 3) == ri) || (pi == 24 + ri);
    for (int ct = 0; ct < (pi < 30 ? 8 : 4); ++ct) {
      if (localpair && ct < 4) continue;
      unsigned e = (unsigned)pi | ((unsigned)ct << 6);
      if (ct < 4 && pi >= 16 && nextc[ct] < 10) e |= (unsigned)(++nextc[ct]) << 9;
      if (pi >= 8 && ct < 4 && nzb < NZB) e |= (unsigned)(++nzb) << 13;
      tiles[nt++] = e;
    }
  }
  if (nt != 256 - NPUSH - NCOLL) { fprintf(stderr, "kernel_launch: bad tile table\n"); return; }
  for (int b = 0; b < NPUSH + NCOLL; ++b) p.tab[b] = 0;
  {
    int nxt[8], left[8];
    for (int l = 0; l < 8; ++l) { nxt[l] = NPUSH + NCOLL + ((l - (NPUSH + NCOLL)) & 7); left[l] = (255 - nxt[l]) / 8 + 1; }
    for (int t = 0; t < nt; ++t) {
      int l = 0; while (left[l] == 0) ++l;
      p.tab[nxt[l]] = tiles[t]; nxt[l] += 8; --left[l];
    }
  }
  ag_gemm_kernel<<<256, 512, 0, stream>>>(p);
}
```
